```python
import numpy as np

kernel_name = "gpu_ag_gemm_m4096_k4096_n16384_bf16_q8"


ABSMAX_RTOL = 0.02

COMPUTE_DTYPE = "bfloat16"
EPILOGUE = "quant_int8"
IN_QUANT = None
OUT_QUANT = "int8"
IN_DTYPES = {"x": "bfloat16", "w_mat": "bfloat16"}
WORLD_SIZE = 8

def _to_bf16(x):
    u = np.ascontiguousarray(x, dtype=np.float32).view(np.uint32)
    return ((u + 0x7FFF + ((u >> 16) & 1)) & 0xFFFF0000).view(np.float32)

def setup_inputs():

    x = _to_bf16(np.random.randn(4096, 4096).astype(np.float32))
    w_mat = _to_bf16((np.random.randn(4096, 16384) / np.sqrt(4096)).astype(np.float32))
    return {"x": x, "w_mat": w_mat}

def _quant_dequant(y, amax):
    scale = amax / 127.0
    q = np.clip(np.round(y / scale), -127.0, 127.0)
    return (q * scale).astype(np.float32)
def _epilogue(y):
    return _quant_dequant(y, np.abs(y).max())

def reference(x, w_mat):
    y = (x @ w_mat).astype(np.float32)
    return _epilogue(y)
```

```cpp
#include <hip/hip_runtime.h>
#include <rccl/rccl.h>
#include <cstdio>
#include <cstdint>

#define HIP_CHECK(expr) do { hipError_t _e = (expr); if (_e != hipSuccess) { \
    fprintf(stderr, "HIP error %s at %s:%d\n", hipGetErrorString(_e), __FILE__, __LINE__); \
    return; } } while (0)

#define OUT_AUX 17
namespace agq {

constexpr int NR = 8;
constexpr int MSH = 512, KD = 4096, NSH = 2048;
constexpr int BM = 128, BN = 256, BK = 64, NT = KD / BK;
constexpr int NWG = 256, NTHR = 512;
constexpr int A_TILE = BM * BK * 2;
constexpr int W_TILE = BK * BN * 2;
constexpr int STAGE = A_TILE + W_TILE;
constexpr int NSTAGE = 3;
constexpr int LDS_SCR = NSTAGE * STAGE;
constexpr int LDS_BYTES = LDS_SCR + 256;
constexpr size_t WS_CTR = 0;
constexpr size_t WS_TMO = 32768;
constexpr size_t WS_FLAG = 65536;
constexpr size_t WS_G1 = WS_FLAG + (size_t)NR * NWG * 64;
constexpr size_t WS_G2 = WS_G1 + (size_t)NWG * 128;
constexpr size_t WS_DBG = WS_G2 + (size_t)NR * 8 * 128;
constexpr size_t WS_X = 1u << 20;
constexpr size_t WS_NEED = WS_X + (size_t)MSH * KD * 2;
constexpr unsigned long long DEADLINE_TICKS = 50000000ull;

typedef short s16x4 __attribute__((ext_vector_type(4)));
typedef short s16x8 __attribute__((ext_vector_type(8)));
typedef float f32x4 __attribute__((ext_vector_type(4)));
typedef unsigned u32x4 __attribute__((ext_vector_type(4)));
#define LAS __attribute__((address_space(3)))
#define GAS __attribute__((address_space(1)))

struct Params {
  const char* x;
  const char* w;
  char* out;
  char* ws[NR];
  int rank;
};

#define STAMP(i) do {} while (0)

template <int IMM>
__device__ __forceinline__ void glds16(const char* base, unsigned voff, unsigned m0v) {
  asm volatile("s_mov_b32 m0, %2\n\ts_nop 0\n\tglobal_load_lds_dwordx4 %0, %1 offset:%3"
               :: "v"(voff), "s"(base), "s"(m0v), "n"(IMM) : "memory");
}
template <int CTRL>
__device__ __forceinline__ unsigned dpp_max(unsigned v) {
  return max(v, (unsigned)__builtin_amdgcn_update_dpp(0, (int)v, CTRL, 0xf, 0xf, true));
}
__device__ __forceinline__ unsigned wave_max(unsigned v) {
  v = dpp_max<0xB1>(v);
  v = dpp_max<0x4E>(v);
  v = dpp_max<0x141>(v);
  v = dpp_max<0x140>(v);
  const unsigned a = __builtin_amdgcn_readlane(v, 0), b = __builtin_amdgcn_readlane(v, 16);
  const unsigned c = __builtin_amdgcn_readlane(v, 32), d = __builtin_amdgcn_readlane(v, 48);
  return max(max(a, b), max(c, d));
}
template <int SCOPE>
__device__ __forceinline__ unsigned wait_granules(const unsigned long long* g, unsigned seq, unsigned* tmo, int lane) {
  const unsigned long long t0 = __builtin_amdgcn_s_memrealtime();
  unsigned long long v;
  for (;;) {
    const unsigned tv = __hip_atomic_load(tmo, __ATOMIC_RELAXED, __HIP_MEMORY_SCOPE_AGENT);
    v = __hip_atomic_load(g, __ATOMIC_RELAXED, SCOPE);
    if (__all((unsigned)(v >> 32) == seq)) break;
    if (__builtin_amdgcn_readfirstlane(tv) == seq || __builtin_amdgcn_s_memrealtime() - t0 > DEADLINE_TICKS) {
      if (lane == 0) __hip_atomic_store(tmo, seq, __ATOMIC_RELAXED, __HIP_MEMORY_SCOPE_AGENT);
      break;
    }
    __builtin_amdgcn_s_sleep(1);
  }
  return wave_max((unsigned)v);
}

__global__ void __launch_bounds__(NTHR, 2) ag_gemm_q8(Params p) {
  extern __shared__ __attribute__((aligned(16))) char smem_raw[];
  LAS char* smem = (LAS char*)smem_raw;
  const int tid = threadIdx.x, lane = tid & 63;
  const int wave = __builtin_amdgcn_readfirstlane(tid >> 6);
  const int bid = blockIdx.x;
  char* const myws = p.ws[p.rank];
  STAMP(0);

  unsigned* const ctr = (unsigned*)(myws + WS_CTR + (size_t)bid * 128);
  const unsigned seq_v = __hip_atomic_load(ctr, __ATOMIC_RELAXED, __HIP_MEMORY_SCOPE_AGENT) + 1u;
  char* const peer_ws = p.ws[lane & 7];
  {
    const int ct = bid >> 2, ch = bid & 3;
    const char* src = p.x + (size_t)(ch * BM + wave * 16 + (lane & 15)) * (KD * 2) + ct * (BK * 2) + (lane >> 4) * 16;
    const u32x4 v0 = *(const u32x4*)src;
    const u32x4 v1 = *(const u32x4*)(src + 64);
    auto rs = __builtin_amdgcn_make_buffer_rsrc(myws + WS_X, 0, MSH * KD * 2, 0x00020000);
    const unsigned o = (unsigned)bid * A_TILE + wave * 2048 + lane * 16;
    __builtin_amdgcn_raw_buffer_store_b128(v0, rs, o, 0, 17);
    __builtin_amdgcn_raw_buffer_store_b128(v1, rs, o + 1024, 0, 17);
  }
  const unsigned seq = __builtin_amdgcn_readfirstlane(seq_v);
  asm volatile("s_waitcnt vmcnt(0)" ::: "memory");
  LAS unsigned* scr = (LAS unsigned*)(smem + LDS_SCR);
  __builtin_amdgcn_s_barrier();
  asm volatile("" ::: "memory");
  STAMP(7);
  if (wave == 0) {
    if (lane < NR)
      __hip_atomic_store((unsigned*)(peer_ws + WS_FLAG + ((size_t)p.rank * NWG + bid) * 64), seq, __ATOMIC_RELAXED, __HIP_MEMORY_SCOPE_SYSTEM);
  }
  if (tid == 64 * 2) __hip_atomic_store(ctr, seq, __ATOMIC_RELAXED, __HIP_MEMORY_SCOPE_AGENT);
  STAMP(1);

  const int shard = bid & 7, idx = bid >> 3, panel = idx & 3, colt = idx >> 2;
  const int wr = wave >> 2, wc = wave & 3;
  const unsigned lds0 = (unsigned)(size_t)smem;
  unsigned wvoff;
  {
    const int r = lane >> 3;
    const int f = ((r >> 1) & 1) * 2 + (wave & 1) * 4;
    wvoff = (wave * 8 + r) * (NSH * 2) + (((lane & 7) ^ f) * 16);
  }
  const char* wbase = p.w + colt * (BN * 2);
  const int ap0 = (wave + 2 * colt) & 15, ap1 = (wave + 8 + 2 * colt) & 15;
  const unsigned avoff0 = ap0 * 1024 + lane * 16, avoff1 = ap1 * 1024 + lane * 16;
  const char* abase = p.ws[shard] + WS_X + (size_t)panel * A_TILE;
  const unsigned wm0 = lds0 + wave * 1024, am0 = lds0 + W_TILE + ap0 * 1024, am1 = lds0 + W_TILE + ap1 * 1024;
#define ISSUE_W(t, so, i) glds16<(i) * 128>(wbase + (size_t)(t) * (BK * NSH * 2), wvoff, wm0 + (so) + (i) * (8192 - 128))
#define ISSUE_A0(t, so) glds16<0>(abase + (size_t)(t) * (4 * A_TILE), avoff0, am0 + (so))
#define ISSUE_A1(t, so) glds16<0>(abase + (size_t)(t) * (4 * A_TILE), avoff1, am1 + (so))

  ISSUE_W(0, 0, 0); ISSUE_W(0, 0, 1); ISSUE_W(0, 0, 2); ISSUE_W(0, 0, 3);
  ISSUE_W(1, STAGE, 0); ISSUE_W(1, STAGE, 1); ISSUE_W(1, STAGE, 2); ISSUE_W(1, STAGE, 3);

  unsigned* tmo = (unsigned*)(myws + WS_TMO);
  if (wave == 0) {
    const unsigned* fl = (const unsigned*)(myws + WS_FLAG + ((size_t)shard * NWG + lane * 4 + panel) * 64);
    const unsigned long long t0 = __builtin_amdgcn_s_memrealtime();
    for (;;) {
      const unsigned tv = __hip_atomic_load(tmo, __ATOMIC_RELAXED, __HIP_MEMORY_SCOPE_AGENT);
      const unsigned fv = __hip_atomic_load(fl, __ATOMIC_RELAXED, __HIP_MEMORY_SCOPE_SYSTEM);
      if (__all(fv >= seq)) break;
      if (__builtin_amdgcn_readfirstlane(tv) == seq || __builtin_amdgcn_s_memrealtime() - t0 > DEADLINE_TICKS) {
        if (lane == 0) __hip_atomic_store(tmo, seq, __ATOMIC_RELAXED, __HIP_MEMORY_SCOPE_AGENT);
        break;
      }
      __builtin_amdgcn_s_sleep(1);
    }
    STAMP(8);
  }
  asm volatile("" ::: "memory");
  __builtin_amdgcn_s_barrier();
  asm volatile("" ::: "memory");
  STAMP(2);
  ISSUE_A0(0, 0); ISSUE_A1(0, 0);
  ISSUE_A0(1, STAGE); ISSUE_A1(1, STAGE);

  unsigned wadr[4];
  {
    const int g = lane >> 4, q = (lane & 15) >> 2, pp = lane & 3;
    const int f = (q >> 1) * 2 + (g & 1) * 4;
#pragma unroll
    for (int j = 0; j < 4; ++j) wadr[j] = wc * 8192 + g * 1024 + q * 128 + (((2 * j + (pp >> 1)) ^ f) * 16) + (pp & 1) * 8;
  }
  const unsigned aadr = W_TILE + wr * 8192 + lane * 16;

  f32x4 acc[4][4];
#pragma unroll
  for (int m = 0; m < 4; ++m)
#pragma unroll
    for (int j = 0; j < 4; ++j) acc[m][j] = (f32x4){0.f, 0.f, 0.f, 0.f};

  asm volatile("s_waitcnt vmcnt(2)" ::: "memory");

  auto tile = [&](int t, int so, int sn, bool issue) {
    s16x8 af[2][4], wf[2][4];
#pragma unroll
    for (int ks = 0; ks < 2; ++ks) {
#pragma unroll
      for (int j = 0; j < 4; ++j) {
        const s16x4 lo = __builtin_amdgcn_ds_read_tr16_b64_v4i16((LAS s16x4*)(smem + so + wadr[j] + ks * 4096));
        const s16x4 hi = __builtin_amdgcn_ds_read_tr16_b64_v4i16((LAS s16x4*)(smem + so + wadr[j] + ks * 4096 + 512));
        wf[ks][j] = __builtin_shufflevector(lo, hi, 0, 1, 2, 3, 4, 5, 6, 7);
        if (j == 0) af[ks][0] = *(const LAS s16x8*)(smem + so + aadr + ks * 1024);
      }
#pragma unroll
      for (int m = 1; m < 4; ++m) af[ks][m] = *(const LAS s16x8*)(smem + so + aadr + m * 2048 + ks * 1024);
      if (issue && ks == 0) ISSUE_W(t + 2, sn, 0);
    }
    __builtin_amdgcn_sched_barrier(0);
#pragma unroll
    for (int ks = 0; ks < 2; ++ks) {
#pragma unroll
      for (int m = 0; m < 4; ++m) {
#pragma unroll
        for (int j = 0; j < 4; ++j)
          acc[m][j] = __builtin_amdgcn_mfma_f32_16x16x32_bf16(wf[ks][j], af[ks][m], acc[m][j], 0, 0, 0);
        __builtin_amdgcn_sched_barrier(0);
        if (issue) {
          const int s = ks * 4 + m;
          if (s == 0) ISSUE_W(t + 2, sn, 1);
          else if (s == 1) ISSUE_W(t + 2, sn, 2);
          else if (s == 2) ISSUE_W(t + 2, sn, 3);
          else if (s == 3) ISSUE_A0(t + 2, sn);
          else if (s == 4) ISSUE_A1(t + 2, sn);
          __builtin_amdgcn_sched_barrier(0);
        }
      }
    }
  };

  int so = 0;
  for (int t = 0; t < NT - 2; ++t) {
    asm volatile("s_waitcnt vmcnt(6) lgkmcnt(0)" ::: "memory");
    __builtin_amdgcn_s_barrier();
    asm volatile("" ::: "memory");
    const int sn = so == 0 ? 2 * STAGE : so - STAGE;
    tile(t, so, sn, true);
    so = so == 2 * STAGE ? 0 : so + STAGE;
  }
  asm volatile("s_waitcnt vmcnt(6) lgkmcnt(0)" ::: "memory");
  __builtin_amdgcn_s_barrier();
  asm volatile("" ::: "memory");
  tile(NT - 2, so, 0, false);
  so = so == 2 * STAGE ? 0 : so + STAGE;
  asm volatile("s_waitcnt vmcnt(0) lgkmcnt(0)" ::: "memory");
  __builtin_amdgcn_s_barrier();
  asm volatile("" ::: "memory");
  tile(NT - 1, so, 0, false);
  STAMP(3);

  float mx = 0.f;
#pragma unroll
  for (int m = 0; m < 4; ++m)
#pragma unroll
    for (int j = 0; j < 4; ++j)
#pragma unroll
      for (int r = 0; r < 4; ++r) mx = fmaxf(mx, fabsf(acc[m][j][r]));
  const unsigned wmx = wave_max(__float_as_uint(mx));
  if (lane == 0) scr[4 + wave] = wmx;
  __syncthreads();
  if (wave == 0) {
    const u32x4 ra = *(const LAS u32x4*)(scr + 4), rb = *(const LAS u32x4*)(scr + 8);
    unsigned v = max(max(max(ra[0], ra[1]), max(ra[2], ra[3])), max(max(rb[0], rb[1]), max(rb[2], rb[3])));
    unsigned long long* g1 = (unsigned long long*)(myws + WS_G1 + (size_t)shard * 32 * 128);
    if (lane == 0) __hip_atomic_store(g1 + idx * 16, ((unsigned long long)seq << 32) | v, __ATOMIC_RELAXED, __HIP_MEMORY_SCOPE_AGENT);
    STAMP(4);
    if (idx == 0) {
      v = wait_granules<__HIP_MEMORY_SCOPE_AGENT>(g1 + (lane & 31) * 16, seq, tmo, lane);
      if (lane < NR)
        __hip_atomic_store((unsigned long long*)(peer_ws + WS_G2 + ((size_t)p.rank * 8 + shard) * 128),
                           ((unsigned long long)seq << 32) | v, __ATOMIC_RELAXED, __HIP_MEMORY_SCOPE_SYSTEM);
      STAMP(9);
    }
    v = wait_granules<__HIP_MEMORY_SCOPE_SYSTEM>((const unsigned long long*)(myws + WS_G2 + (size_t)lane * 128), seq, tmo, lane);
    const bool timed_out = __builtin_amdgcn_readfirstlane(__hip_atomic_load(tmo, __ATOMIC_RELAXED, __HIP_MEMORY_SCOPE_AGENT)) == seq;
    if (lane == 0) { scr[12] = v; scr[13] = timed_out; }
  }
  __syncthreads();
  const float amax = __uint_as_float(scr[12]);
  const unsigned bad = scr[13];
  STAMP(5);

  const float scale = amax / 127.0f;
  const float inv = scale > 0.f ? 1.0f / scale : 0.f;
  char* orow = p.out + (size_t)(shard * MSH + panel * BM + wr * 64 + (lane & 15)) * NSH + colt * BN + wc * 64 + (lane >> 4) * 16;
  auto ors = __builtin_amdgcn_make_buffer_rsrc(p.out, 0, NR * MSH * NSH, 0x00020000);
  const unsigned ooff = (unsigned)(orow - p.out);
#pragma unroll
  for (int m = 0; m < 4; ++m) {
    unsigned d[4];
#pragma unroll
    for (int j = 0; j < 4; ++j) {
      unsigned b[4];
#pragma unroll
      for (int r = 0; r < 4; ++r) {
        const float q = __builtin_amdgcn_fmed3f(acc[m][j][r] * inv, -127.0f, 127.0f);
        b[r] = __float_as_uint(q + 12582912.0f);
      }
      const unsigned lo = __builtin_amdgcn_perm(b[1], b[0], 0x0c0c0400u);
      const unsigned hi = __builtin_amdgcn_perm(b[3], b[2], 0x04000c0cu);
      d[j] = lo | hi;
    }
    auto s0 = __builtin_amdgcn_permlane32_swap(d[0], d[2], false, false);
    auto s1 = __builtin_amdgcn_permlane32_swap(d[1], d[3], false, false);
    auto u0 = __builtin_amdgcn_permlane16_swap(s0[0], s1[0], false, false);
    auto u1 = __builtin_amdgcn_permlane16_swap(s0[1], s1[1], false, false);
    u32x4 o = {u0[0], u0[1], u1[0], u1[1]};
    if (bad) o = (u32x4){0x80808080u, 0x80808080u, 0x80808080u, 0x80808080u};
    __builtin_amdgcn_raw_buffer_store_b128(o, ors, ooff + m * 16 * NSH, 0, OUT_AUX);
  }
  STAMP(6);
}

}

extern "C" void kernel_launch(int rank, int nranks, ncclComm_t comm,
                              hipStream_t stream,
                              void* const* d_in, const int* in_sizes, int n_in,
                              void* d_out, int out_size,
                              const int* in_dtypes, int out_dtype,
                              void* const* d_out_all,
                              void* const* d_ws_all, size_t ws_bytes) {
  using namespace agq;
  if (nranks != NR || (unsigned)rank >= (unsigned)NR || n_in != 2 || ws_bytes < WS_NEED ||
      in_sizes[0] != MSH * KD || in_sizes[1] != KD * NSH || out_size != NR * MSH * NSH) {
    fprintf(stderr, "ag_gemm_q8: unexpected problem shape\n");
    return;
  }
  static bool configured = false;
  if (!configured) {
    HIP_CHECK(hipFuncSetAttribute((const void*)ag_gemm_q8, hipFuncAttributeMaxDynamicSharedMemorySize, LDS_BYTES));
    configured = true;
  }
  Params p;
  p.x = (const char*)d_in[0];
  p.w = (const char*)d_in[1];
  p.out = (char*)d_out;
  for (int i = 0; i < NR; ++i) p.ws[i] = (char*)d_ws_all[i];
  p.rank = rank;
  hipLaunchKernelGGL(ag_gemm_q8, dim3(NWG), dim3(NTHR), LDS_BYTES, stream, p);
  HIP_CHECK(hipGetLastError());
}
```

```cpp
#include <hip/hip_runtime.h>
#include <rccl/rccl.h>
#include <cstdio>
#include <cstdint>

#define HIP_CHECK(expr) do { hipError_t _e = (expr); if (_e != hipSuccess) { \
    fprintf(stderr, "HIP error %s at %s:%d\n", hipGetErrorString(_e), __FILE__, __LINE__); \
    return; } } while (0)

#define OUT_AUX 17
#define PORD 3
#define XAUX 17
#define TLS 3
#define NEARLY 2
namespace agq {

constexpr int NR = 8;
constexpr int MSH = 512, KD = 4096, NSH = 2048;
constexpr int BM = 128, BN = 256, BK = 64, NT = KD / BK;
constexpr int NWG = 256, NTHR = 512;
constexpr int A_TILE = BM * BK * 2;
constexpr int W_TILE = BK * BN * 2;
constexpr int STAGE = A_TILE + W_TILE;
constexpr int NSTAGE = 3;
constexpr int LDS_SCR = NSTAGE * STAGE;
constexpr int LDS_BYTES = LDS_SCR + 512;
constexpr size_t WS_CTR = 0;
constexpr size_t WS_TMO = 32768;
constexpr size_t WS_FLAG = 65536;
constexpr size_t WS_G1 = WS_FLAG + (size_t)NR * NWG * 64;
constexpr size_t WS_G2 = WS_G1 + (size_t)NWG * 128;
constexpr size_t WS_DBG = WS_G2 + (size_t)NR * 8 * 128;
constexpr size_t WS_X = 1u << 20;
constexpr unsigned XSTRIDE = 4 * A_TILE;
constexpr size_t WS_NEED = WS_X + (size_t)NT * XSTRIDE;
static_assert(WS_DBG + NWG * 128 <= WS_X, "layout");
constexpr unsigned long long DEADLINE_TICKS = 50000000ull;

typedef short s16x4 __attribute__((ext_vector_type(4)));
typedef short s16x8 __attribute__((ext_vector_type(8)));
typedef float f32x4 __attribute__((ext_vector_type(4)));
typedef float f32x2 __attribute__((ext_vector_type(2)));
typedef short s16x2 __attribute__((ext_vector_type(2)));
typedef unsigned u32x4 __attribute__((ext_vector_type(4)));
#define LAS __attribute__((address_space(3)))
#define GAS __attribute__((address_space(1)))

typedef GAS char gchar;
typedef GAS unsigned gu32;
typedef GAS unsigned long long gu64;

struct Params {
  const char* x;
  const char* w;
  char* out;
  char* myws;
  char* ws[NR];
  int rank;
};

#define STAMPW(i) do {} while (0)
#define STAMP(i) do {} while (0)

template <int IMM>
__device__ __forceinline__ void glds16(const gchar* base, unsigned voff, unsigned m0v) {
  asm volatile("s_mov_b32 m0, %2\n\ts_nop 0\n\tglobal_load_lds_dwordx4 %0, %1 offset:%3"
               :: "v"(voff), "s"(base), "s"(m0v), "n"(IMM) : "memory");
}
template <int CTRL>
__device__ __forceinline__ unsigned dpp_max(unsigned v) {
  return max(v, (unsigned)__builtin_amdgcn_update_dpp(0, (int)v, CTRL, 0xf, 0xf, true));
}
__device__ __forceinline__ unsigned wave_max(unsigned v) {
  v = dpp_max<0xB1>(v);
  v = dpp_max<0x4E>(v);
  v = dpp_max<0x141>(v);
  v = dpp_max<0x140>(v);
  const unsigned a = __builtin_amdgcn_readlane(v, 0), b = __builtin_amdgcn_readlane(v, 16);
  const unsigned c = __builtin_amdgcn_readlane(v, 32), d = __builtin_amdgcn_readlane(v, 48);
  return max(max(a, b), max(c, d));
}
__device__ __forceinline__ bool give_up(unsigned tv, unsigned seq, unsigned long long t0, gu32* tmo, int lane) {
  if (__builtin_amdgcn_readfirstlane(tv) == seq || __builtin_amdgcn_s_memrealtime() - t0 > DEADLINE_TICKS) {
    if (lane == 0) __hip_atomic_store(tmo, seq, __ATOMIC_RELAXED, __HIP_MEMORY_SCOPE_AGENT);
    return true;
  }
  __builtin_amdgcn_s_sleep(1);
  return false;
}
template <int N, int SCOPE>
__device__ __forceinline__ unsigned wait_granules(const gu64* g, unsigned seq, gu32* tmo, int lane) {
  const unsigned long long t0 = __builtin_amdgcn_s_memrealtime();
  unsigned mx;
  for (;;) {
    const unsigned tv = __hip_atomic_load(tmo, __ATOMIC_RELAXED, __HIP_MEMORY_SCOPE_AGENT);
    bool ok = true;
    mx = 0;
#pragma unroll
    for (int i = 0; i < N; ++i) {
      const unsigned long long v = __hip_atomic_load(g + i, __ATOMIC_RELAXED, SCOPE);
      ok &= (unsigned)(v >> 32) == seq;
      mx = max(mx, (unsigned)v);
    }
    if (__all(ok)) break;
    if (give_up(tv, seq, t0, tmo, lane)) break;
  }
  return wave_max(mx);
}

__global__ void __launch_bounds__(NTHR, 2) ag_gemm_q8(Params p) {
  extern __shared__ __attribute__((aligned(16))) char smem_raw[];
  LAS char* smem = (LAS char*)smem_raw;
  const int tid = threadIdx.x, lane = tid & 63;
  const int wave = __builtin_amdgcn_readfirstlane(tid >> 6);
  const int bid = blockIdx.x;
  gchar* myws = (gchar*)p.myws;
  gchar* sws = (gchar*)p.ws[bid & 7];
  const gchar* px = (const gchar*)p.x;
  const gchar* pw = (const gchar*)p.w;
  gchar* pout = (gchar*)p.out;
  int rank = p.rank;
  gchar* wsw = (gchar*)p.ws[wave];
  asm volatile("" : "+s"(wsw));
  asm volatile("" : "+s"(myws), "+s"(sws), "+s"(px), "+s"(pw), "+s"(pout), "+s"(rank));
  STAMP(0);

  const int shard = bid & 7, idx = bid >> 3, panel = idx & 3, colt = idx >> 2;
  const int wr = wave >> 2, wc = wave & 3;
  const unsigned lds0 = (unsigned)(size_t)smem;
  unsigned wvoff;
  {
    const int r = lane >> 3;
    const int f = ((r >> 1) & 1) * 2 + (wave & 1) * 4;
    wvoff = (wave * 8 + r) * (NSH * 2) + (((lane & 7) ^ f) * 16);
  }
  const gchar* wbase = pw + colt * (BN * 2);
  const int arb = (wave + colt) & 7;
  const unsigned avoff = arb * 2048 + lane * 16;
  const gchar* abase = sws + WS_X + (size_t)panel * A_TILE;
  const unsigned wm0 = lds0 + wave * 1024, am0 = lds0 + W_TILE + arb * 2048;
#define ISSUE_W(wp, so, i) glds16<(i) * 128>(wp, wvoff, wm0 + (so) + (i) * (8192 - 128))
#define ISSUE_A(ap, so, h) glds16<(h) * 1024>(ap, avoff, am0 + (so))
  LAS unsigned* scr = (LAS unsigned*)(smem + LDS_SCR);

  const int cpan = bid & 3, crb = (bid >> 2) & 7, ckg = bid >> 5;
  const bool early = ckg < NEARLY;
  gu32* const ctr = (gu32*)(myws + WS_CTR + (size_t)bid * 128);
  gu32* const myflag = (gu32*)(wsw + WS_FLAG + ((size_t)rank * NWG + bid) * 64);
  const gchar* xsrc = px + (size_t)(cpan * BM + crb * 16 + (lane & 15)) * (KD * 2) + (ckg * 8 + wave) * (BK * 2) + (lane >> 4) * 16;
  const auto xrs = __builtin_amdgcn_make_buffer_rsrc((void*)(myws + WS_X), 0, NT * XSTRIDE, 0x00020000);
  const unsigned xoff = (ckg * 8 + wave) * XSTRIDE + cpan * A_TILE + crb * 2048 + lane * 16;
  unsigned seq_v;
  u32x4 v0, v1;
  if (early) {
    asm volatile("global_load_dword %0, %3, %4 sc1\n\tglobal_load_dwordx4 %1, %5, off\n\tglobal_load_dwordx4 %2, %5, off offset:64\n\t"
                 "s_waitcnt vmcnt(0)"
                 : "=&v"(seq_v), "=&v"(v0), "=&v"(v1) : "v"(0), "s"(ctr), "v"(xsrc) : "memory");
    STAMP(10);
    __builtin_amdgcn_raw_buffer_store_b128(v0, xrs, xoff, 0, XAUX);
    __builtin_amdgcn_raw_buffer_store_b128(v1, xrs, xoff + 1024, 0, XAUX);
    asm volatile("s_waitcnt vmcnt(0)" ::: "memory");
    __builtin_amdgcn_s_barrier();
    asm volatile("" ::: "memory");
    if (lane == 0) __hip_atomic_store(myflag, __builtin_amdgcn_readfirstlane(seq_v) + 1u, __ATOMIC_RELAXED, __HIP_MEMORY_SCOPE_SYSTEM);
    STAMP(1);
    ISSUE_W(wbase, 0, 0); ISSUE_W(wbase, 0, 1); ISSUE_W(wbase, 0, 2); ISSUE_W(wbase, 0, 3);
  } else {
    asm volatile("global_load_dword %0, %1, %2 sc1" : "=&v"(seq_v) : "v"(0), "s"(ctr) : "memory");
    ISSUE_W(wbase, 0, 0); ISSUE_W(wbase, 0, 1); ISSUE_W(wbase, 0, 2); ISSUE_W(wbase, 0, 3);
    asm volatile("s_waitcnt vmcnt(4)" : "+v"(seq_v) :: "memory");
  }
  const unsigned seq = __builtin_amdgcn_readfirstlane(seq_v) + 1u;

  gu32* tmo = (gu32*)(myws + WS_TMO);
  const gu32* fl = (const gu32*)(myws + WS_FLAG + ((size_t)shard * NWG + lane * 4 + panel) * 64);
  auto wait_flags = [&](int nlanes) {
    const unsigned long long t0 = __builtin_amdgcn_s_memrealtime();
    for (;;) {
      const unsigned tv = __hip_atomic_load(tmo, __ATOMIC_RELAXED, __HIP_MEMORY_SCOPE_AGENT);
      const unsigned fv = __hip_atomic_load(fl, __ATOMIC_RELAXED, __HIP_MEMORY_SCOPE_SYSTEM);
      if (__all(fv >= seq || lane >= nlanes)) break;
      if (give_up(tv, seq, t0, tmo, lane)) break;
    }
  };
  gchar* peer_ws = (gchar*)p.ws[lane & 7];
  if (wave == 0) { wait_flags(8 * NEARLY); asm volatile("" : "+v"(peer_ws)); STAMP(8); }
  asm volatile("" ::: "memory");
  __builtin_amdgcn_s_barrier();
  asm volatile("" ::: "memory");
  STAMP(2);
  if (tid == 64 * 2) __hip_atomic_store(ctr, seq, __ATOMIC_RELAXED, __HIP_MEMORY_SCOPE_AGENT);
  ISSUE_A(abase, 0, 0); ISSUE_A(abase, 0, 1);
  __builtin_amdgcn_s_barrier();
  {
    const gchar* w1 = wbase + BK * NSH * 2;
    ISSUE_W(w1, STAGE, 0); ISSUE_W(w1, STAGE, 1);
    ISSUE_W(w1, STAGE, 2); ISSUE_W(w1, STAGE, 3);
  }
  ISSUE_A(abase + XSTRIDE, STAGE, 0); ISSUE_A(abase + XSTRIDE, STAGE, 1);

  unsigned wadr[4];
  {
    const int g = lane >> 4, q = (lane & 15) >> 2, pp = lane & 3;
    const int f = (q >> 1) * 2 + (g & 1) * 4;
#pragma unroll
    for (int j = 0; j < 4; ++j) wadr[j] = wc * 8192 + g * 1024 + q * 128 + (((2 * j + (pp >> 1)) ^ f) * 16) + (pp & 1) * 8;
  }
  const unsigned aadr = W_TILE + wr * 8192 + lane * 16;

  f32x4 acc[4][4];
#pragma unroll
  for (int m = 0; m < 4; ++m)
#pragma unroll
    for (int j = 0; j < 4; ++j) acc[m][j] = (f32x4){0.f, 0.f, 0.f, 0.f};

  s16x8 af[2][4], wf[2][4];
  auto read_w = [&](int so, int ks) {
#pragma unroll
    for (int j = 0; j < 4; ++j) {
      const s16x4 lo = __builtin_amdgcn_ds_read_tr16_b64_v4i16((LAS s16x4*)(smem + so + wadr[j] + ks * 4096));
      const s16x4 hi = __builtin_amdgcn_ds_read_tr16_b64_v4i16((LAS s16x4*)(smem + so + wadr[j] + ks * 4096 + 512));
      wf[ks][j] = __builtin_shufflevector(lo, hi, 0, 1, 2, 3, 4, 5, 6, 7);
    }
  };
  auto read_a = [&](int so, int ks) {
#pragma unroll
    for (int m = 0; m < 4; ++m) af[ks][m] = *(const LAS s16x8*)(smem + so + aadr + m * 2048 + ks * 1024);
  };
  auto mma = [&](int ks, int m) {
#pragma unroll
    for (int j = 0; j < 4; ++j)
      acc[m][j] = __builtin_amdgcn_mfma_f32_16x16x32_bf16(wf[ks][j], af[ks][m], acc[m][j], 0, 0, 0);
    __builtin_amdgcn_sched_barrier(0);
  };
#define TILE_SYNC(n) do { asm volatile("s_waitcnt vmcnt(" #n ") lgkmcnt(0)" ::: "memory"); __builtin_amdgcn_s_barrier(); asm volatile("" ::: "memory"); } while (0)

  TILE_SYNC(6);
  STAMP(11);
  int so = 0;
  const int tl = early ? -8 : TLS * (ckg - NEARLY) + 1;
  for (int t = 0; t < NT - 2; ++t) {
    const int dt = t - tl;
    if (dt == 4) {
      if (lane == 0) __hip_atomic_store(myflag, seq, __ATOMIC_RELAXED, __HIP_MEMORY_SCOPE_SYSTEM);
    }
    if (wave == 0 && t < 32) {
      if ((t & 15) == 12) {
        asm volatile("s_mov_b32 m0, %1\n\ts_nop 0\n\tglobal_load_lds_dword %0, off sc0 sc1" :: "v"(fl), "s"(lds0 + LDS_SCR + 256) : "memory");
      } else if ((t & 15) == 13) {
        const unsigned pv = scr[64 + lane];
        const int nl = t < 16 ? 32 : 64;
        if (!__all(pv >= seq || lane >= nl)) wait_flags(nl);
      }
    }
    const gchar* ap = abase + (size_t)(t + 2) * XSTRIDE;
    const gchar* wp = wbase + (size_t)(t + 2) * (BK * NSH * 2);
    const int sn = so == 0 ? 2 * STAGE : so - STAGE;
    read_w(so, 0); read_a(so, 0);
    ISSUE_W(wp, sn, 0);
    read_w(so, 1); read_a(so, 1);
    __builtin_amdgcn_sched_barrier(0);
    mma(0, 0); ISSUE_W(wp, sn, 1); __builtin_amdgcn_sched_barrier(0);
    mma(0, 1); ISSUE_W(wp, sn, 2); __builtin_amdgcn_sched_barrier(0);
    mma(0, 2); ISSUE_W(wp, sn, 3); __builtin_amdgcn_sched_barrier(0);
    mma(0, 3); ISSUE_A(ap, sn, 0); __builtin_amdgcn_sched_barrier(0);
    mma(1, 0); ISSUE_A(ap, sn, 1); __builtin_amdgcn_sched_barrier(0);
    mma(1, 1); mma(1, 2); mma(1, 3);
    so = so == 2 * STAGE ? 0 : so + STAGE;
    if (dt == 0) {
      asm volatile("global_load_dwordx4 v[240:243], %0, off\n\tglobal_load_dwordx4 v[244:247], %0, off offset:64" :: "v"(xsrc)
                   : "memory", "v240", "v241", "v242", "v243", "v244", "v245", "v246", "v247");
      TILE_SYNC(8);
    } else if (dt == 2) {
      asm volatile("global_store_dwordx4 %0, v[240:243], %1 sc0 sc1\n\tglobal_store_dwordx4 %0, v[244:247], %1 offset:1024 sc0 sc1\n\ts_nop 1"
                   :: "v"(xoff), "s"(myws + WS_X) : "memory");
      TILE_SYNC(8);
    } else {
      TILE_SYNC(6);
    }
  }
  read_w(so, 0); read_a(so, 0); read_w(so, 1); read_a(so, 1);
  __builtin_amdgcn_sched_barrier(0);
  mma(0, 0); mma(0, 1); mma(0, 2); mma(0, 3); mma(1, 0); mma(1, 1); mma(1, 2); mma(1, 3);
  so = so == 2 * STAGE ? 0 : so + STAGE;
  TILE_SYNC(1);
  read_w(so, 0); read_a(so, 0); read_w(so, 1);
  __builtin_amdgcn_sched_barrier(0);
  mma(0, 0); mma(0, 1); mma(0, 2); mma(0, 3);
  TILE_SYNC(0);
  read_a(so, 1);
  __builtin_amdgcn_sched_barrier(0);
  mma(1, 0); mma(1, 1); mma(1, 2); mma(1, 3);
  STAMP(3);

  float mx = 0.f;
#pragma unroll
  for (int m = 0; m < 4; ++m)
#pragma unroll
    for (int j = 0; j < 4; ++j)
#pragma unroll
      for (int r = 0; r < 4; ++r) mx = fmaxf(mx, fabsf(acc[m][j][r]));
  const unsigned wmx = wave_max(__float_as_uint(mx));
  if (lane == 0) scr[4 + wave] = wmx;
  __syncthreads();
  if (wave == 0) {
    const u32x4 ra = *(const LAS u32x4*)(scr + 4), rb = *(const LAS u32x4*)(scr + 8);
    const unsigned gm = max(max(max(ra[0], ra[1]), max(ra[2], ra[3])), max(max(rb[0], rb[1]), max(rb[2], rb[3])));
    gu64* g1 = (gu64*)(myws + WS_G1 + (size_t)shard * 32 * 128);
    if (lane == 0) __hip_atomic_store(g1 + idx * 16, ((unsigned long long)seq << 32) | gm, __ATOMIC_RELAXED, __HIP_MEMORY_SCOPE_AGENT);
    STAMP(4);
    if (idx == 0) {
      const unsigned v = wait_granules<1, __HIP_MEMORY_SCOPE_AGENT>(g1 + (lane & 31) * 16, seq, tmo, lane);
      if (lane < NR)
        __hip_atomic_store((gu64*)(peer_ws + WS_G2 + ((size_t)rank * 8 + shard) * 128),
                           ((unsigned long long)seq << 32) | v, __ATOMIC_RELAXED, __HIP_MEMORY_SCOPE_SYSTEM);
      STAMP(9);
    }
    const unsigned v = wait_granules<1, __HIP_MEMORY_SCOPE_SYSTEM>((const gu64*)(myws + WS_G2 + (size_t)lane * 128), seq, tmo, lane);
    const bool timed_out = __builtin_amdgcn_readfirstlane(__hip_atomic_load(tmo, __ATOMIC_RELAXED, __HIP_MEMORY_SCOPE_AGENT)) == seq;
    const float amax = __uint_as_float(v);
    if (lane == 0) { scr[0] = __float_as_uint(amax > 0.f ? 127.0f / amax : 0.f); scr[1] = timed_out; }
  }
  __syncthreads();
  const float inv = __uint_as_float(scr[0]);
  const unsigned bad = scr[1];
  STAMP(5);

  auto ors = __builtin_amdgcn_make_buffer_rsrc((void*)pout, 0, NR * MSH * NSH, 0x00020000);
  const unsigned ooff = (shard * MSH + panel * BM + wr * 64 + (lane & 15)) * NSH + colt * BN + wc * 64 + (lane >> 4) * 16;
  const f32x2 inv2 = {inv, inv}, magic = {12582912.0f, 12582912.0f};
#pragma unroll
  for (int m = 0; m < 4; ++m) {
    unsigned d[4];
#pragma unroll
    for (int j = 0; j < 4; ++j) {
      s16x2 h[2];
#pragma unroll
      for (int r = 0; r < 2; ++r) {
        const f32x2 y = {acc[m][j][2 * r], acc[m][j][2 * r + 1]};
        const f32x2 q = __builtin_elementwise_fma(y, inv2, magic);
        const unsigned pk = __builtin_amdgcn_perm(__float_as_uint(q[1]), __float_as_uint(q[0]), 0x05040100u);
        s16x2 c = __builtin_bit_cast(s16x2, pk);
        c = __builtin_elementwise_min(c, (s16x2){127, 127});
        h[r] = __builtin_elementwise_max(c, (s16x2){-127, -127});
      }
      d[j] = __builtin_amdgcn_perm(__builtin_bit_cast(unsigned, h[1]), __builtin_bit_cast(unsigned, h[0]), 0x06040200u);
    }
    auto s0 = __builtin_amdgcn_permlane32_swap(d[0], d[2], false, false);
    auto s1 = __builtin_amdgcn_permlane32_swap(d[1], d[3], false, false);
    auto u0 = __builtin_amdgcn_permlane16_swap(s0[0], s1[0], false, false);
    auto u1 = __builtin_amdgcn_permlane16_swap(s0[1], s1[1], false, false);
    u32x4 o = {u0[0], u0[1], u1[0], u1[1]};
    if (bad) o = (u32x4){0x80808080u, 0x80808080u, 0x80808080u, 0x80808080u};
    __builtin_amdgcn_raw_buffer_store_b128(o, ors, ooff + m * 16 * NSH, 0, OUT_AUX);
  }
  STAMP(6);
}

}

extern "C" void kernel_launch(int rank, int nranks, ncclComm_t comm,
                              hipStream_t stream,
                              void* const* d_in, const int* in_sizes, int n_in,
                              void* d_out, int out_size,
                              const int* in_dtypes, int out_dtype,
                              void* const* d_out_all,
                              void* const* d_ws_all, size_t ws_bytes) {
  using namespace agq;
  if (nranks != NR || (unsigned)rank >= (unsigned)NR || n_in != 2 || ws_bytes < WS_NEED ||
      in_sizes[0] != MSH * KD || in_sizes[1] != KD * NSH || out_size != NR * MSH * NSH) {
    fprintf(stderr, "ag_gemm_q8: unexpected problem shape\n");
    return;
  }
  static bool configured = false;
  if (!configured) {
    HIP_CHECK(hipFuncSetAttribute((const void*)ag_gemm_q8, hipFuncAttributeMaxDynamicSharedMemorySize, LDS_BYTES));
    configured = true;
  }
  Params p;
  p.x = (const char*)d_in[0];
  p.w = (const char*)d_in[1];
  p.out = (char*)d_out;
  for (int i = 0; i < NR; ++i) p.ws[i] = (char*)d_ws_all[i];
  p.myws = (char*)d_ws_all[rank];
  p.rank = rank;
  hipLaunchKernelGGL(ag_gemm_q8, dim3(NWG), dim3(NTHR), LDS_BYTES, stream, p);
  HIP_CHECK(hipGetLastError());
}
```

```cpp
#include <hip/hip_runtime.h>
#include <rccl/rccl.h>
#include <cstdio>
#include <cstdint>

#define HIP_CHECK(expr) do { hipError_t _e = (expr); if (_e != hipSuccess) { \
    fprintf(stderr, "HIP error %s at %s:%d\n", hipGetErrorString(_e), __FILE__, __LINE__); \
    return; } } while (0)

#define OUT_AUX 17
#define PORD 3
#define XAUX 17
#define POLL_FAST 12
#define POLL_SLEEP 4
#define PD 0
#define SPEC_T 8
#define TLS 3
#define NEARLY 2
namespace agq {

constexpr int NR = 8;
constexpr int MSH = 512, KD = 4096, NSH = 2048;
constexpr int BM = 128, BN = 256, BK = 64, NT = KD / BK;
constexpr int NWG = 256, NTHR = 512;
constexpr int A_TILE = BM * BK * 2;
constexpr int W_TILE = BK * BN * 2;
constexpr int STAGE = A_TILE + W_TILE;
constexpr int NSTAGE = 3;
constexpr int LDS_SCR = NSTAGE * STAGE;
constexpr int LDS_BYTES = LDS_SCR + 768;
constexpr size_t WS_CTR = 0;
constexpr size_t WS_TMO = 32768;
constexpr size_t WS_FLAG = 65536;
constexpr size_t WS_G1 = WS_FLAG + (size_t)NR * NWG * 64;
constexpr size_t WS_G2 = WS_G1 + (size_t)NWG * 128;
constexpr size_t WS_DBG = WS_G2 + (size_t)NR * 8 * 128;
constexpr size_t WS_X = 1u << 20;
constexpr unsigned XSTRIDE = 4 * A_TILE;
constexpr size_t WS_NEED = WS_X + (size_t)NT * XSTRIDE;
static_assert(WS_DBG + NWG * 128 <= WS_X, "layout");
constexpr unsigned long long DEADLINE_TICKS = 50000000ull;

typedef short s16x4 __attribute__((ext_vector_type(4)));
typedef short s16x8 __attribute__((ext_vector_type(8)));
typedef float f32x4 __attribute__((ext_vector_type(4)));
typedef float f32x2 __attribute__((ext_vector_type(2)));
typedef short s16x2 __attribute__((ext_vector_type(2)));
typedef unsigned u32x4 __attribute__((ext_vector_type(4)));
#define LAS __attribute__((address_space(3)))
#define GAS __attribute__((address_space(1)))

typedef GAS char gchar;
typedef GAS unsigned gu32;
typedef GAS unsigned long long gu64;

struct Params {
  const char* x;
  const char* w;
  char* out;
  char* myws;
  char* ws[NR];
  int rank;
};

#define STAMPW(i) do {} while (0)
#define STAMP(i) do {} while (0)

template <int IMM>
__device__ __forceinline__ void glds16(const gchar* base, unsigned voff, unsigned m0v) {
  asm volatile("s_mov_b32 m0, %2\n\ts_nop 0\n\tglobal_load_lds_dwordx4 %0, %1 offset:%3"
               :: "v"(voff), "s"(base), "s"(m0v), "n"(IMM) : "memory");
}
template <int CTRL>
__device__ __forceinline__ unsigned dpp_max(unsigned v) {
  return max(v, (unsigned)__builtin_amdgcn_update_dpp(0, (int)v, CTRL, 0xf, 0xf, true));
}
__device__ __forceinline__ unsigned wave_max(unsigned v) {
  v = dpp_max<0xB1>(v);
  v = dpp_max<0x4E>(v);
  v = dpp_max<0x141>(v);
  v = dpp_max<0x140>(v);
  const unsigned a = __builtin_amdgcn_readlane(v, 0), b = __builtin_amdgcn_readlane(v, 16);
  const unsigned c = __builtin_amdgcn_readlane(v, 32), d = __builtin_amdgcn_readlane(v, 48);
  return max(max(a, b), max(c, d));
}
__device__ __forceinline__ bool give_up(unsigned seq, unsigned long long t0, gu32* tmo, int lane) {
  const unsigned tv = __hip_atomic_load(tmo, __ATOMIC_RELAXED, __HIP_MEMORY_SCOPE_AGENT);
  if (__builtin_amdgcn_readfirstlane(tv) == seq || __builtin_amdgcn_s_memrealtime() - t0 > DEADLINE_TICKS) {
    if (lane == 0) __hip_atomic_store(tmo, seq, __ATOMIC_RELAXED, __HIP_MEMORY_SCOPE_AGENT);
    return true;
  }
  return false;
}
template <int SCOPE>
__device__ __forceinline__ int wait_granules(const gu64* g, unsigned long long& v, int allow, unsigned seq, gu32* tmo, int lane,
                                             bool& timed_out) {
  const unsigned long long t0 = __builtin_amdgcn_s_memrealtime();
  for (unsigned n = 1;; ++n) {
    const unsigned tv = __hip_atomic_load(tmo, __ATOMIC_RELAXED, __HIP_MEMORY_SCOPE_AGENT);
    if ((unsigned)(v >> 32) != seq) v = __hip_atomic_load(g, __ATOMIC_RELAXED, SCOPE);
    const int miss = __builtin_popcountll(__ballot((unsigned)(v >> 32) != seq));
    timed_out = __builtin_amdgcn_readfirstlane(tv) == seq;
    if (miss <= allow || timed_out) return miss;
    if (miss > allow + POLL_FAST) __builtin_amdgcn_s_sleep(POLL_SLEEP);
    if ((n & 15) == 0 && give_up(seq, t0, tmo, lane)) { timed_out = true; return miss; }
  }
}
__device__ __forceinline__ unsigned granule_max(unsigned long long v, unsigned seq) {
  return wave_max((unsigned)(v >> 32) == seq ? (unsigned)v : 0u);
}

__global__ void __launch_bounds__(NTHR, 2) ag_gemm_q8(Params p) {
  extern __shared__ __attribute__((aligned(16))) char smem_raw[];
  LAS char* smem = (LAS char*)smem_raw;
  const int tid = threadIdx.x, lane = tid & 63;
  const int wave = __builtin_amdgcn_readfirstlane(tid >> 6);
  const int bid = blockIdx.x;
  gchar* myws = (gchar*)p.myws;
  gchar* sws = (gchar*)p.ws[bid & 7];
  const gchar* px = (const gchar*)p.x;
  const gchar* pw = (const gchar*)p.w;
  gchar* pout = (gchar*)p.out;
  int rank = p.rank;
  gchar* wsw = (gchar*)p.ws[wave];
  asm volatile("" : "+s"(wsw));
  asm volatile("" : "+s"(myws), "+s"(sws), "+s"(px), "+s"(pw), "+s"(pout), "+s"(rank));
  STAMP(0);

  const int shard = bid & 7, idx = bid >> 3, panel = idx & 3, colt = idx >> 2;
  const int wr = wave >> 2, wc = wave & 3;
  const unsigned lds0 = (unsigned)(size_t)smem;
  unsigned wvoff;
  {
    const int r = lane >> 3;
    const int f = ((r >> 1) & 1) * 2 + (wave & 1) * 4;
    wvoff = (wave * 8 + r) * (NSH * 2) + (((lane & 7) ^ f) * 16);
  }
  const gchar* wbase = pw + colt * (BN * 2);
  const int arb = (wave + colt) & 7;
  const unsigned avoff = arb * 2048 + lane * 16;
  const gchar* abase = sws + WS_X + (size_t)panel * A_TILE;
  const unsigned wm0 = lds0 + wave * 1024, am0 = lds0 + W_TILE + arb * 2048;
#define ISSUE_W(wp, so, i) glds16<(i) * 128>(wp, wvoff, wm0 + (so) + (i) * (8192 - 128))
#define ISSUE_A(ap, so, h) glds16<(h) * 1024>(ap, avoff, am0 + (so))
  LAS unsigned* scr = (LAS unsigned*)(smem + LDS_SCR);
#define ISSUE_PF(wp) do {} while (0)
#define NPF 0

  const int cpan = bid & 3, crb = (bid >> 2) & 7, ckg = bid >> 5;
  const bool early = ckg < NEARLY;
  gu32* const ctr = (gu32*)(myws + WS_CTR + (size_t)bid * 128);
  gu32* const myflag = (gu32*)(wsw + WS_FLAG + ((size_t)rank * NWG + bid) * 64);
  const gchar* xsrc = px + (size_t)(cpan * BM + crb * 16 + (lane & 15)) * (KD * 2) + (ckg * 8 + wave) * (BK * 2) + (lane >> 4) * 16;
  const auto xrs = __builtin_amdgcn_make_buffer_rsrc((void*)(myws + WS_X), 0, NT * XSTRIDE, 0x00020000);
  const unsigned xoff = (ckg * 8 + wave) * XSTRIDE + cpan * A_TILE + crb * 2048 + lane * 16;
  unsigned seq_v;
  u32x4 v0, v1;
  if (early) {
    asm volatile("global_load_dword %0, %3, %4 sc1\n\tglobal_load_dwordx4 %1, %5, off\n\tglobal_load_dwordx4 %2, %5, off offset:64\n\t"
                 "s_waitcnt vmcnt(0)"
                 : "=&v"(seq_v), "=&v"(v0), "=&v"(v1) : "v"(0), "s"(ctr), "v"(xsrc) : "memory");
    __builtin_amdgcn_raw_buffer_store_b128(v0, xrs, xoff, 0, XAUX);
    __builtin_amdgcn_raw_buffer_store_b128(v1, xrs, xoff + 1024, 0, XAUX);
    asm volatile("s_waitcnt vmcnt(0)" ::: "memory");
    __builtin_amdgcn_s_barrier();
    asm volatile("" ::: "memory");
    if (lane == 0) __hip_atomic_store(myflag, __builtin_amdgcn_readfirstlane(seq_v) + 1u, __ATOMIC_RELAXED, __HIP_MEMORY_SCOPE_SYSTEM);
    STAMP(1);
    ISSUE_W(wbase, 0, 0); ISSUE_W(wbase, 0, 1); ISSUE_W(wbase, 0, 2); ISSUE_W(wbase, 0, 3);
  } else {
    asm volatile("global_load_dword %0, %1, %2 sc1" : "=&v"(seq_v) : "v"(0), "s"(ctr) : "memory");
    ISSUE_W(wbase, 0, 0); ISSUE_W(wbase, 0, 1); ISSUE_W(wbase, 0, 2); ISSUE_W(wbase, 0, 3);
    asm volatile("s_waitcnt vmcnt(4)" : "+v"(seq_v) :: "memory");
  }
  const unsigned seq = __builtin_amdgcn_readfirstlane(seq_v) + 1u;

  gu32* tmo = (gu32*)(myws + WS_TMO);
  const gu32* fl = (const gu32*)(myws + WS_FLAG + ((size_t)shard * NWG + lane * 4 + panel) * 64);
  auto wait_flags = [&](int nlanes) {
    const unsigned long long t0 = __builtin_amdgcn_s_memrealtime();
    unsigned fv = lane < nlanes ? 0u : seq;
    for (unsigned n = 1;; ++n) {
      if (fv < seq) fv = __hip_atomic_load(fl, __ATOMIC_RELAXED, __HIP_MEMORY_SCOPE_SYSTEM);
      if (__all(fv >= seq)) break;
      if ((n & 15) == 0 && give_up(seq, t0, tmo, lane)) break;
    }
  };
  gchar* peer_ws = (gchar*)p.ws[lane & 7];
  if (wave == 0) { wait_flags(8 * NEARLY); asm volatile("" : "+v"(peer_ws)); STAMP(8); }
  asm volatile("" ::: "memory");
  __builtin_amdgcn_s_barrier();
  asm volatile("" ::: "memory");
  STAMP(2);
  if (tid == 64 * 2) __hip_atomic_store(ctr, seq, __ATOMIC_RELAXED, __HIP_MEMORY_SCOPE_AGENT);
  ISSUE_A(abase, 0, 0); ISSUE_A(abase, 0, 1);
  __builtin_amdgcn_s_barrier();
  {
    const gchar* w1 = wbase + BK * NSH * 2;
    ISSUE_W(w1, STAGE, 0); ISSUE_W(w1, STAGE, 1);
    ISSUE_W(w1, STAGE, 2); ISSUE_W(w1, STAGE, 3);
  }
  ISSUE_A(abase + XSTRIDE, STAGE, 0); ISSUE_A(abase + XSTRIDE, STAGE, 1);

  unsigned wadr[4];
  {
    const int g = lane >> 4, q = (lane & 15) >> 2, pp = lane & 3;
    const int f = (q >> 1) * 2 + (g & 1) * 4;
#pragma unroll
    for (int j = 0; j < 4; ++j) wadr[j] = wc * 8192 + g * 1024 + q * 128 + (((2 * j + (pp >> 1)) ^ f) * 16) + (pp & 1) * 8;
  }
  const unsigned aadr = W_TILE + wr * 8192 + lane * 16;

  f32x4 acc[4][4];
#pragma unroll
  for (int m = 0; m < 4; ++m)
#pragma unroll
    for (int j = 0; j < 4; ++j) acc[m][j] = (f32x4){0.f, 0.f, 0.f, 0.f};

  s16x8 af[2][4], wf[2][4];
  auto read_w = [&](int so, int ks) {
#pragma unroll
    for (int j = 0; j < 4; ++j) {
      const s16x4 lo = __builtin_amdgcn_ds_read_tr16_b64_v4i16((LAS s16x4*)(smem + so + wadr[j] + ks * 4096));
      const s16x4 hi = __builtin_amdgcn_ds_read_tr16_b64_v4i16((LAS s16x4*)(smem + so + wadr[j] + ks * 4096 + 512));
      wf[ks][j] = __builtin_shufflevector(lo, hi, 0, 1, 2, 3, 4, 5, 6, 7);
    }
  };
  auto read_a = [&](int so, int ks) {
#pragma unroll
    for (int m = 0; m < 4; ++m) af[ks][m] = *(const LAS s16x8*)(smem + so + aadr + m * 2048 + ks * 1024);
  };
  auto mma = [&](int ks, int m) {
#pragma unroll
    for (int j = 0; j < 4; ++j)
      acc[m][j] = __builtin_amdgcn_mfma_f32_16x16x32_bf16(wf[ks][j], af[ks][m], acc[m][j], 0, 0, 0);
    __builtin_amdgcn_sched_barrier(0);
  };
#define TILE_SYNC(n) do { asm volatile("s_waitcnt vmcnt(%0) lgkmcnt(0)" :: "n"(n) : "memory"); __builtin_amdgcn_s_barrier(); asm volatile("" ::: "memory"); } while (0)

  TILE_SYNC(6);
  STAMP(11);
  int so = 0;
  const int tl = early ? -8 : TLS * (ckg - NEARLY) + 1;
  for (int t = 0; t < NT - 2; ++t) {
    const int dt = t - tl;
    if (dt == 4) {
      if (lane == 0) __hip_atomic_store(myflag, seq, __ATOMIC_RELAXED, __HIP_MEMORY_SCOPE_SYSTEM);
    }
    if (wave == 0 && t < 32) {
      if ((t & 15) == 12) {
        asm volatile("s_mov_b32 m0, %1\n\ts_nop 0\n\tglobal_load_lds_dword %0, off sc0 sc1" :: "v"(fl), "s"(lds0 + LDS_SCR + 256) : "memory");
      } else if ((t & 15) == 13) {
        const unsigned pv = scr[64 + lane];
        const int nl = t < 16 ? 32 : 64;
        if (!__all(pv >= seq || lane >= nl)) wait_flags(nl);
      }
    }
    const gchar* ap = abase + (size_t)(t + 2) * XSTRIDE;
    const gchar* wp = wbase + (size_t)(t + 2) * (BK * NSH * 2);
    const int sn = so == 0 ? 2 * STAGE : so - STAGE;
    read_w(so, 0); read_a(so, 0);
    ISSUE_W(wp, sn, 0);
    read_w(so, 1); read_a(so, 1);
    __builtin_amdgcn_sched_barrier(0);
    mma(0, 0); ISSUE_W(wp, sn, 1); __builtin_amdgcn_sched_barrier(0);
    mma(0, 1); ISSUE_W(wp, sn, 2); __builtin_amdgcn_sched_barrier(0);
    mma(0, 2); ISSUE_W(wp, sn, 3); __builtin_amdgcn_sched_barrier(0);
    mma(0, 3); ISSUE_A(ap, sn, 0); __builtin_amdgcn_sched_barrier(0);
    mma(1, 0); ISSUE_A(ap, sn, 1); __builtin_amdgcn_sched_barrier(0);
    mma(1, 1); ISSUE_PF(wbase + (size_t)min(t + 2 + PD, NT - 1) * (BK * NSH * 2)); __builtin_amdgcn_sched_barrier(0);
    mma(1, 2); mma(1, 3);
    so = so == 2 * STAGE ? 0 : so + STAGE;
    if (dt == 0) {
      asm volatile("global_load_dwordx4 v[240:243], %0, off\n\tglobal_load_dwordx4 v[244:247], %0, off offset:64" :: "v"(xsrc)
                   : "memory", "v240", "v241", "v242", "v243", "v244", "v245", "v246", "v247");
      TILE_SYNC(8 + NPF);
    } else if (dt == 2) {
      asm volatile("global_store_dwordx4 %0, v[240:243], %1 sc0 sc1\n\tglobal_store_dwordx4 %0, v[244:247], %1 offset:1024 sc0 sc1\n\ts_nop 1"
                   :: "v"(xoff), "s"(myws + WS_X) : "memory");
      TILE_SYNC(8 + NPF);
    } else {
      TILE_SYNC(6 + NPF);
    }
  }
  read_w(so, 0); read_a(so, 0); read_w(so, 1); read_a(so, 1);
  __builtin_amdgcn_sched_barrier(0);
  mma(0, 0); mma(0, 1); mma(0, 2); mma(0, 3); mma(1, 0); mma(1, 1); mma(1, 2); mma(1, 3);
  so = so == 2 * STAGE ? 0 : so + STAGE;
  TILE_SYNC(1 + NPF);
  read_w(so, 0); read_a(so, 0); read_w(so, 1);
  __builtin_amdgcn_sched_barrier(0);
  mma(0, 0); mma(0, 1); mma(0, 2); mma(0, 3);
  TILE_SYNC(NPF);
  read_a(so, 1);
  __builtin_amdgcn_sched_barrier(0);
  mma(1, 0); mma(1, 1); mma(1, 2); mma(1, 3);
  STAMP(3);

  float mx = 0.f;
#pragma unroll
  for (int m = 0; m < 4; ++m)
#pragma unroll
    for (int j = 0; j < 4; ++j)
#pragma unroll
      for (int r = 0; r < 4; ++r) mx = fmaxf(mx, fabsf(acc[m][j][r]));
  const unsigned wmx = wave_max(__float_as_uint(mx));
  if (lane == 0) scr[4 + wave] = wmx;
  __syncthreads();
  const gu64* g2 = (const gu64*)(myws + WS_G2 + (size_t)lane * 128);
  unsigned long long gv = 0;
  unsigned cur = 0;
  if (wave == 0) {
    const u32x4 ra = *(const LAS u32x4*)(scr + 4), rb = *(const LAS u32x4*)(scr + 8);
    const unsigned gm = max(max(max(ra[0], ra[1]), max(ra[2], ra[3])), max(max(rb[0], rb[1]), max(rb[2], rb[3])));
    gu64* g1 = (gu64*)(myws + WS_G1 + (size_t)shard * 32 * 128);
    if (lane == 0) __hip_atomic_store(g1 + idx * 16, ((unsigned long long)seq << 32) | gm, __ATOMIC_RELAXED, __HIP_MEMORY_SCOPE_AGENT);
    STAMP(4);
    bool timed_out;
    if (idx == 0) {
      unsigned long long v = 0;
      wait_granules<__HIP_MEMORY_SCOPE_AGENT>(g1 + (lane & 31) * 16, v, 0, seq, tmo, lane, timed_out);
      const unsigned gmax = granule_max(v, seq);
      if (lane < NR && !timed_out)
        __hip_atomic_store((gu64*)(peer_ws + WS_G2 + ((size_t)rank * 8 + shard) * 128),
                           ((unsigned long long)seq << 32) | gmax, __ATOMIC_RELAXED, __HIP_MEMORY_SCOPE_SYSTEM);
      STAMP(9);
    }
    const int miss = wait_granules<__HIP_MEMORY_SCOPE_SYSTEM>(g2, gv, SPEC_T, seq, tmo, lane, timed_out);
    cur = max(granule_max(gv, seq), gm);
    const float amax = __uint_as_float(cur);
    if (lane == 0) { scr[0] = __float_as_uint(amax > 0.f ? 127.0f / amax : 0.f); scr[1] = timed_out; scr[2] = miss == 0 || timed_out; }
  }
  __syncthreads();
  float inv = __uint_as_float(scr[0]);
  unsigned bad = scr[1];
  unsigned last = __builtin_amdgcn_readfirstlane(scr[2]);
  STAMP(7);

  auto ors = __builtin_amdgcn_make_buffer_rsrc((void*)pout, 0, NR * MSH * NSH, 0x00020000);
  const unsigned ooff = (shard * MSH + panel * BM + wr * 64 + (lane & 15)) * NSH + colt * BN + wc * 64 + (lane >> 4) * 16;
  const f32x2 magic = {12582912.0f, 12582912.0f};
#pragma unroll 1
  for (;;) {
    const f32x2 inv2 = {inv, inv};
#pragma unroll
    for (int m = 0; m < 4; ++m) {
      unsigned d[4];
#pragma unroll
      for (int j = 0; j < 4; ++j) {
        s16x2 h[2];
#pragma unroll
        for (int r = 0; r < 2; ++r) {
          const f32x2 y = {acc[m][j][2 * r], acc[m][j][2 * r + 1]};
          const f32x2 q = __builtin_elementwise_fma(y, inv2, magic);
          const unsigned pk = __builtin_amdgcn_perm(__float_as_uint(q[1]), __float_as_uint(q[0]), 0x05040100u);
          s16x2 c = __builtin_bit_cast(s16x2, pk);
          c = __builtin_elementwise_min(c, (s16x2){127, 127});
          h[r] = __builtin_elementwise_max(c, (s16x2){-127, -127});
        }
        d[j] = __builtin_amdgcn_perm(__builtin_bit_cast(unsigned, h[1]), __builtin_bit_cast(unsigned, h[0]), 0x06040200u);
      }
      auto s0 = __builtin_amdgcn_permlane32_swap(d[0], d[2], false, false);
      auto s1 = __builtin_amdgcn_permlane32_swap(d[1], d[3], false, false);
      auto u0 = __builtin_amdgcn_permlane16_swap(s0[0], s1[0], false, false);
      auto u1 = __builtin_amdgcn_permlane16_swap(s0[1], s1[1], false, false);
      u32x4 o = {u0[0], u0[1], u1[0], u1[1]};
      if (bad) o = (u32x4){0x80808080u, 0x80808080u, 0x80808080u, 0x80808080u};
      __builtin_amdgcn_raw_buffer_store_b128(o, ors, ooff + m * 16 * NSH, 0, OUT_AUX);
    }
    if (last) break;
    if (wave == 0) {
      bool timed_out;
      wait_granules<__HIP_MEMORY_SCOPE_SYSTEM>(g2, gv, 0, seq, tmo, lane, timed_out);
      const unsigned fin = granule_max(gv, seq);
      const float amax = __uint_as_float(fin);
      if (lane == 0) { scr[12] = __float_as_uint(amax > 0.f ? 127.0f / amax : 0.f); scr[13] = timed_out; scr[14] = fin != cur || timed_out; }
    }
    __syncthreads();
    STAMP(5);
    if (!__builtin_amdgcn_readfirstlane(scr[14])) break;
    inv = __uint_as_float(scr[12]);
    bad = scr[13];
    last = 1;
    asm volatile("s_waitcnt vmcnt(0)" ::: "memory");
    STAMP(10);
  }
  STAMP(6);
}

}

extern "C" void kernel_launch(int rank, int nranks, ncclComm_t comm,
                              hipStream_t stream,
                              void* const* d_in, const int* in_sizes, int n_in,
                              void* d_out, int out_size,
                              const int* in_dtypes, int out_dtype,
                              void* const* d_out_all,
                              void* const* d_ws_all, size_t ws_bytes) {
  using namespace agq;
  if (nranks != NR || (unsigned)rank >= (unsigned)NR || n_in != 2 || ws_bytes < WS_NEED ||
      in_sizes[0] != MSH * KD || in_sizes[1] != KD * NSH || out_size != NR * MSH * NSH) {
    fprintf(stderr, "ag_gemm_q8: unexpected problem shape\n");
    return;
  }
  static bool configured = false;
  if (!configured) {
    HIP_CHECK(hipFuncSetAttribute((const void*)ag_gemm_q8, hipFuncAttributeMaxDynamicSharedMemorySize, LDS_BYTES));
    configured = true;
  }
  Params p;
  p.x = (const char*)d_in[0];
  p.w = (const char*)d_in[1];
  p.out = (char*)d_out;
  for (int i = 0; i < NR; ++i) p.ws[i] = (char*)d_ws_all[i];
  p.myws = (char*)d_ws_all[rank];
  p.rank = rank;
  hipLaunchKernelGGL(ag_gemm_q8, dim3(NWG), dim3(NTHR), LDS_BYTES, stream, p);
  HIP_CHECK(hipGetLastError());
}
```

```cpp
#include <hip/hip_runtime.h>
#include <rccl/rccl.h>
#include <cstdio>
#include <cstdint>

#define HIP_CHECK(expr) do { hipError_t _e = (expr); if (_e != hipSuccess) { \
    fprintf(stderr, "HIP error %s at %s:%d\n", hipGetErrorString(_e), __FILE__, __LINE__); \
    return; } } while (0)

namespace agq {

constexpr int NR = 8;
constexpr int MSH = 512, KD = 4096, NSH = 2048;
constexpr int BM = 128, BN = 256, BK = 64, NT = KD / BK;
constexpr int NWG = 256, NTHR = 512;
constexpr int A_TILE = BM * BK * 2;
constexpr int W_TILE = BK * BN * 2;
constexpr int STAGE = A_TILE + W_TILE;
constexpr int NSTAGE = 3;
constexpr int LDS_SCR = NSTAGE * STAGE;
constexpr int LDS_BYTES = LDS_SCR + 512;
constexpr int NEARLY = 2;
constexpr int SPEC_T = 8;
constexpr size_t WS_CTR = 0;
constexpr size_t WS_TMO = 32768;
constexpr size_t WS_FLAG = 65536;
constexpr size_t WS_G1 = WS_FLAG + (size_t)NR * NWG * 64;
constexpr size_t WS_G2 = WS_G1 + (size_t)NWG * 128;
constexpr size_t WS_DBG = WS_G2 + (size_t)NR * 8 * 128;
constexpr size_t WS_X = 1u << 20;
constexpr unsigned XSTRIDE = 4 * A_TILE;
constexpr size_t WS_NEED = WS_X + (size_t)NT * XSTRIDE;
static_assert(WS_DBG + NWG * 128 <= WS_X, "layout");
constexpr unsigned long long DEADLINE_TICKS = 50000000ull;

typedef short s16x4 __attribute__((ext_vector_type(4)));
typedef short s16x8 __attribute__((ext_vector_type(8)));
typedef float f32x4 __attribute__((ext_vector_type(4)));
typedef float f32x2 __attribute__((ext_vector_type(2)));
typedef short s16x2 __attribute__((ext_vector_type(2)));
typedef unsigned u32x4 __attribute__((ext_vector_type(4)));
#define LAS __attribute__((address_space(3)))
#define GAS __attribute__((address_space(1)))

typedef GAS char gchar;
typedef GAS unsigned gu32;
typedef GAS unsigned long long gu64;

struct Params {
  const char* x;
  const char* w;
  char* out;
  char* myws;
  char* ws[NR];
  int rank;
};

#define STAMPW(i) do {} while (0)
#define STAMP(i) do {} while (0)

template <int IMM>
__device__ __forceinline__ void glds16(const gchar* base, unsigned voff, unsigned m0v) {
  asm volatile("s_mov_b32 m0, %2\n\ts_nop 0\n\tglobal_load_lds_dwordx4 %0, %1 offset:%3"
               :: "v"(voff), "s"(base), "s"(m0v), "n"(IMM) : "memory");
}
template <int CTRL>
__device__ __forceinline__ unsigned dpp_max(unsigned v) {
  return max(v, (unsigned)__builtin_amdgcn_update_dpp(0, (int)v, CTRL, 0xf, 0xf, true));
}
__device__ __forceinline__ unsigned wave_max(unsigned v) {
  v = dpp_max<0xB1>(v);
  v = dpp_max<0x4E>(v);
  v = dpp_max<0x141>(v);
  v = dpp_max<0x140>(v);
  const unsigned a = __builtin_amdgcn_readlane(v, 0), b = __builtin_amdgcn_readlane(v, 16);
  const unsigned c = __builtin_amdgcn_readlane(v, 32), d = __builtin_amdgcn_readlane(v, 48);
  return max(max(a, b), max(c, d));
}
__device__ __forceinline__ bool give_up(unsigned seq, unsigned long long t0, gu32* tmo, int lane) {
  const unsigned tv = __hip_atomic_load(tmo, __ATOMIC_RELAXED, __HIP_MEMORY_SCOPE_AGENT);
  if (__builtin_amdgcn_readfirstlane(tv) == seq || __builtin_amdgcn_s_memrealtime() - t0 > DEADLINE_TICKS) {
    if (lane == 0) __hip_atomic_store(tmo, seq, __ATOMIC_RELAXED, __HIP_MEMORY_SCOPE_AGENT);
    return true;
  }
  return false;
}
template <int SCOPE>
__device__ __forceinline__ int wait_granules(const gu64* g, unsigned long long& v, int allow, unsigned seq, gu32* tmo, int lane,
                                             bool& timed_out) {
  const unsigned long long t0 = __builtin_amdgcn_s_memrealtime();
  for (unsigned n = 1;; ++n) {
    const unsigned tv = __hip_atomic_load(tmo, __ATOMIC_RELAXED, __HIP_MEMORY_SCOPE_AGENT);
    if ((unsigned)(v >> 32) != seq) v = __hip_atomic_load(g, __ATOMIC_RELAXED, SCOPE);
    const int miss = __builtin_popcountll(__ballot((unsigned)(v >> 32) != seq));
    timed_out = __builtin_amdgcn_readfirstlane(tv) == seq;
    if (miss <= allow || timed_out) return miss;
    if (miss > allow + 12) __builtin_amdgcn_s_sleep(4);
    if ((n & 15) == 0 && give_up(seq, t0, tmo, lane)) { timed_out = true; return miss; }
  }
}
__device__ __forceinline__ unsigned granule_max(unsigned long long v, unsigned seq) {
  return wave_max((unsigned)(v >> 32) == seq ? (unsigned)v : 0u);
}

__global__ void __launch_bounds__(NTHR, 2) ag_gemm_q8(Params p) {
  extern __shared__ __attribute__((aligned(16))) char smem_raw[];
  LAS char* smem = (LAS char*)smem_raw;
  const int tid = threadIdx.x, lane = tid & 63;
  const int wave = __builtin_amdgcn_readfirstlane(tid >> 6);
  const int bid = blockIdx.x;
  gchar* myws = (gchar*)p.myws;
  gchar* sws = (gchar*)p.ws[bid & 7];
  const gchar* px = (const gchar*)p.x;
  const gchar* pw = (const gchar*)p.w;
  gchar* pout = (gchar*)p.out;
  int rank = p.rank;
  gchar* wsw = (gchar*)p.ws[wave];
  asm volatile("" : "+s"(wsw));
  asm volatile("" : "+s"(myws), "+s"(sws), "+s"(px), "+s"(pw), "+s"(pout), "+s"(rank));
  STAMP(0);

  const int shard = bid & 7, idx = bid >> 3, panel = idx & 3, colt = idx >> 2;
  const int wr = wave >> 2, wc = wave & 3;
  const unsigned lds0 = (unsigned)(size_t)smem;
  unsigned wvoff;
  {
    const int r = lane >> 3;
    const int f = ((r >> 1) & 1) * 2 + (wave & 1) * 4;
    wvoff = (wave * 8 + r) * (NSH * 2) + (((lane & 7) ^ f) * 16);
  }
  const gchar* wbase = pw + colt * (BN * 2);
  const int arb = (wave + colt) & 7;
  const unsigned avoff = arb * 2048 + lane * 16;
  const gchar* abase = sws + WS_X + (size_t)panel * A_TILE;
  const unsigned wm0 = lds0 + wave * 1024, am0 = lds0 + W_TILE + arb * 2048;
#define ISSUE_W(wp, so, i) glds16<(i) * 128>(wp, wvoff, wm0 + (so) + (i) * (8192 - 128))
#define ISSUE_A(ap, so, h) glds16<(h) * 1024>(ap, avoff, am0 + (so))
  LAS unsigned* scr = (LAS unsigned*)(smem + LDS_SCR);

  const int cpan = bid & 3, crb = (bid >> 2) & 7, ckg = bid >> 5;
  const bool early = ckg < NEARLY;
  gu32* const ctr = (gu32*)(myws + WS_CTR + (size_t)bid * 128);
  gu32* const myflag = (gu32*)(wsw + WS_FLAG + ((size_t)rank * NWG + bid) * 64);
  const gchar* xsrc = px + (size_t)(cpan * BM + crb * 16 + (lane & 15)) * (KD * 2) + (ckg * 8 + wave) * (BK * 2) + (lane >> 4) * 16;
  const auto xrs = __builtin_amdgcn_make_buffer_rsrc((void*)(myws + WS_X), 0, NT * XSTRIDE, 0x00020000);
  const unsigned xoff = (ckg * 8 + wave) * XSTRIDE + cpan * A_TILE + crb * 2048 + lane * 16;
  unsigned seq_v;
  u32x4 v0, v1;
  if (early) {
    asm volatile("global_load_dword %0, %3, %4 sc1\n\tglobal_load_dwordx4 %1, %5, off\n\tglobal_load_dwordx4 %2, %5, off offset:64\n\t"
                 "s_waitcnt vmcnt(0)"
                 : "=&v"(seq_v), "=&v"(v0), "=&v"(v1) : "v"(0), "s"(ctr), "v"(xsrc) : "memory");
    __builtin_amdgcn_raw_buffer_store_b128(v0, xrs, xoff, 0, 17);
    __builtin_amdgcn_raw_buffer_store_b128(v1, xrs, xoff + 1024, 0, 17);
    asm volatile("s_waitcnt vmcnt(0)" ::: "memory");
    __builtin_amdgcn_s_barrier();
    asm volatile("" ::: "memory");
    if (lane == 0) __hip_atomic_store(myflag, __builtin_amdgcn_readfirstlane(seq_v) + 1u, __ATOMIC_RELAXED, __HIP_MEMORY_SCOPE_SYSTEM);
    STAMP(1);
    ISSUE_W(wbase, 0, 0); ISSUE_W(wbase, 0, 1); ISSUE_W(wbase, 0, 2); ISSUE_W(wbase, 0, 3);
  } else {
    asm volatile("global_load_dword %0, %1, %2 sc1" : "=&v"(seq_v) : "v"(0), "s"(ctr) : "memory");
    ISSUE_W(wbase, 0, 0); ISSUE_W(wbase, 0, 1); ISSUE_W(wbase, 0, 2); ISSUE_W(wbase, 0, 3);
    asm volatile("s_waitcnt vmcnt(4)" : "+v"(seq_v) :: "memory");
  }
  const unsigned seq = __builtin_amdgcn_readfirstlane(seq_v) + 1u;

  gu32* tmo = (gu32*)(myws + WS_TMO);
  const gu32* fl = (const gu32*)(myws + WS_FLAG + ((size_t)shard * NWG + lane * 4 + panel) * 64);
  auto wait_flags = [&](int nlanes) {
    const unsigned long long t0 = __builtin_amdgcn_s_memrealtime();
    unsigned fv = lane < nlanes ? 0u : seq;
    for (unsigned n = 1;; ++n) {
      if (fv < seq) fv = __hip_atomic_load(fl, __ATOMIC_RELAXED, __HIP_MEMORY_SCOPE_SYSTEM);
      if (__all(fv >= seq)) break;
      if ((n & 15) == 0 && give_up(seq, t0, tmo, lane)) break;
    }
  };
  gchar* peer_ws = (gchar*)p.ws[lane & 7];
  if (wave == 0) { wait_flags(8 * NEARLY); asm volatile("" : "+v"(peer_ws)); STAMP(8); }
  asm volatile("" ::: "memory");
  __builtin_amdgcn_s_barrier();
  asm volatile("" ::: "memory");
  STAMP(2);
  if (tid == 64 * 2) __hip_atomic_store(ctr, seq, __ATOMIC_RELAXED, __HIP_MEMORY_SCOPE_AGENT);
  ISSUE_A(abase, 0, 0); ISSUE_A(abase, 0, 1);
  __builtin_amdgcn_s_barrier();
  {
    const gchar* w1 = wbase + BK * NSH * 2;
    ISSUE_W(w1, STAGE, 0); ISSUE_W(w1, STAGE, 1); ISSUE_W(w1, STAGE, 2); ISSUE_W(w1, STAGE, 3);
  }
  ISSUE_A(abase + XSTRIDE, STAGE, 0); ISSUE_A(abase + XSTRIDE, STAGE, 1);

  unsigned wadr[4];
  {
    const int g = lane >> 4, q = (lane & 15) >> 2, pp = lane & 3;
    const int f = (q >> 1) * 2 + (g & 1) * 4;
#pragma unroll
    for (int j = 0; j < 4; ++j) wadr[j] = wc * 8192 + g * 1024 + q * 128 + (((2 * j + (pp >> 1)) ^ f) * 16) + (pp & 1) * 8;
  }
  const unsigned aadr = W_TILE + wr * 8192 + lane * 16;

  f32x4 acc[4][4];
#pragma unroll
  for (int m = 0; m < 4; ++m)
#pragma unroll
    for (int j = 0; j < 4; ++j) acc[m][j] = (f32x4){0.f, 0.f, 0.f, 0.f};

  s16x8 af[2][4], wf[2][4];
  auto read_w = [&](int so, int ks) {
#pragma unroll
    for (int j = 0; j < 4; ++j) {
      const s16x4 lo = __builtin_amdgcn_ds_read_tr16_b64_v4i16((LAS s16x4*)(smem + so + wadr[j] + ks * 4096));
      const s16x4 hi = __builtin_amdgcn_ds_read_tr16_b64_v4i16((LAS s16x4*)(smem + so + wadr[j] + ks * 4096 + 512));
      wf[ks][j] = __builtin_shufflevector(lo, hi, 0, 1, 2, 3, 4, 5, 6, 7);
    }
  };
  auto read_a = [&](int so, int ks) {
#pragma unroll
    for (int m = 0; m < 4; ++m) af[ks][m] = *(const LAS s16x8*)(smem + so + aadr + m * 2048 + ks * 1024);
  };
  auto mma = [&](int ks, int m) {
#pragma unroll
    for (int j = 0; j < 4; ++j)
      acc[m][j] = __builtin_amdgcn_mfma_f32_16x16x32_bf16(wf[ks][j], af[ks][m], acc[m][j], 0, 0, 0);
    __builtin_amdgcn_sched_barrier(0);
  };
#define TILE_SYNC(n) do { asm volatile("s_waitcnt vmcnt(%0) lgkmcnt(0)" :: "n"(n) : "memory"); __builtin_amdgcn_s_barrier(); asm volatile("" ::: "memory"); } while (0)

  TILE_SYNC(6);
  STAMP(11);
  const int tl = early ? -8 : 3 * (ckg - NEARLY) + 1;
#define SB() __builtin_amdgcn_sched_barrier(0)
  {
    const gchar* wp = wbase + 2 * (BK * NSH * 2);
    read_w(0, 0); read_a(0, 0);
    ISSUE_W(wp, 2 * STAGE, 0); ISSUE_W(wp, 2 * STAGE, 1);
    read_w(0, 1); read_a(0, 1);
    SB();
    mma(0, 0); mma(0, 1); mma(0, 2); mma(0, 3);
    ISSUE_W(wp, 2 * STAGE, 2); ISSUE_W(wp, 2 * STAGE, 3);
    ISSUE_A(abase + 2 * XSTRIDE, 2 * STAGE, 0); ISSUE_A(abase + 2 * XSTRIDE, 2 * STAGE, 1);
    TILE_SYNC(6);
  }
  int sf = 0;
  for (int t = 0; t < NT - 3; ++t) {
    const int dt = t - tl;
    if (dt == 4) {
      if (lane == 0) __hip_atomic_store(myflag, seq, __ATOMIC_RELAXED, __HIP_MEMORY_SCOPE_SYSTEM);
    }
    if (wave == 0 && t < 32) {
      if ((t & 15) == 11) {
        asm volatile("s_mov_b32 m0, %1\n\ts_nop 0\n\tglobal_load_lds_dword %0, off sc0 sc1" :: "v"(fl), "s"(lds0 + LDS_SCR + 256) : "memory");
      } else if ((t & 15) == 12) {
        const unsigned pv = scr[64 + lane];
        const int nl = t < 16 ? 32 : 64;
        if (!__all(pv >= seq || lane >= nl)) wait_flags(nl);
      }
    }
    const gchar* ap = abase + (size_t)(t + 3) * XSTRIDE;
    const gchar* wp = wbase + (size_t)(t + 3) * (BK * NSH * 2);
    const int so = sf == 2 * STAGE ? 0 : sf + STAGE;
    read_w(so, 0); read_a(so, 0);
    SB();
    mma(1, 0); ISSUE_W(wp, sf, 0); SB();
    mma(1, 1); mma(1, 2); ISSUE_W(wp, sf, 1); SB();
    mma(1, 3);
    read_w(so, 1); read_a(so, 1);
    SB();
    mma(0, 0); ISSUE_W(wp, sf, 2); SB();
    mma(0, 1); ISSUE_W(wp, sf, 3); SB();
    mma(0, 2); ISSUE_A(ap, sf, 0); SB();
    mma(0, 3); ISSUE_A(ap, sf, 1);
    sf = so;
    if (dt == 0) {
      asm volatile("global_load_dwordx4 v[240:243], %0, off\n\tglobal_load_dwordx4 v[244:247], %0, off offset:64" :: "v"(xsrc)
                   : "memory", "v240", "v241", "v242", "v243", "v244", "v245", "v246", "v247");
      TILE_SYNC(8);
    } else if (dt == 2) {
      asm volatile("global_store_dwordx4 %0, v[240:243], %1 sc0 sc1\n\tglobal_store_dwordx4 %0, v[244:247], %1 offset:1024 sc0 sc1\n\ts_nop 1"
                   :: "v"(xoff), "s"(myws + WS_X) : "memory");
      TILE_SYNC(8);
    } else {
      TILE_SYNC(6);
    }
  }
  int so = sf == 2 * STAGE ? 0 : sf + STAGE;
  read_w(so, 0); read_a(so, 0);
  SB();
  mma(1, 0); mma(1, 1); mma(1, 2); mma(1, 3);
  read_w(so, 1); read_a(so, 1);
  SB();
  mma(0, 0); mma(0, 1); mma(0, 2); mma(0, 3); mma(1, 0); mma(1, 1); mma(1, 2); mma(1, 3);
  so = so == 2 * STAGE ? 0 : so + STAGE;
  TILE_SYNC(1);
  read_w(so, 0); read_a(so, 0); read_w(so, 1);
  __builtin_amdgcn_sched_barrier(0);
  mma(0, 0); mma(0, 1); mma(0, 2); mma(0, 3);
  TILE_SYNC(0);
  read_a(so, 1);
  __builtin_amdgcn_sched_barrier(0);
  mma(1, 0); mma(1, 1); mma(1, 2); mma(1, 3);
  STAMP(3);

  float mx = 0.f;
#pragma unroll
  for (int m = 0; m < 4; ++m)
#pragma unroll
    for (int j = 0; j < 4; ++j)
#pragma unroll
      for (int r = 0; r < 4; ++r) mx = fmaxf(mx, fabsf(acc[m][j][r]));
  const unsigned wmx = wave_max(__float_as_uint(mx));
  if (lane == 0) scr[4 + wave] = wmx;
  __syncthreads();
  const gu64* g2 = (const gu64*)(myws + WS_G2 + (size_t)lane * 128);
  unsigned long long gv = 0;
  unsigned cur = 0;
  if (wave == 0) {
    const u32x4 ra = *(const LAS u32x4*)(scr + 4), rb = *(const LAS u32x4*)(scr + 8);
    const unsigned gm = max(max(max(ra[0], ra[1]), max(ra[2], ra[3])), max(max(rb[0], rb[1]), max(rb[2], rb[3])));
    gu64* g1 = (gu64*)(myws + WS_G1 + (size_t)shard * 32 * 128);
    if (lane == 0) __hip_atomic_store(g1 + idx * 16, ((unsigned long long)seq << 32) | gm, __ATOMIC_RELAXED, __HIP_MEMORY_SCOPE_AGENT);
    STAMP(4);
    bool timed_out;
    if (idx == 0) {
      unsigned long long v = 0;
      wait_granules<__HIP_MEMORY_SCOPE_AGENT>(g1 + (lane & 31) * 16, v, 0, seq, tmo, lane, timed_out);
      const unsigned gmax = granule_max(v, seq);
      if (lane < NR && !timed_out)
        __hip_atomic_store((gu64*)(peer_ws + WS_G2 + ((size_t)rank * 8 + shard) * 128),
                           ((unsigned long long)seq << 32) | gmax, __ATOMIC_RELAXED, __HIP_MEMORY_SCOPE_SYSTEM);
      STAMP(9);
    }
    const int miss = wait_granules<__HIP_MEMORY_SCOPE_SYSTEM>(g2, gv, SPEC_T, seq, tmo, lane, timed_out);
    cur = max(granule_max(gv, seq), gm);
    const float amax = __uint_as_float(cur);
    if (lane == 0) { scr[0] = __float_as_uint(amax > 0.f ? 127.0f / amax : 0.f); scr[1] = timed_out; scr[2] = miss == 0 || timed_out; }
  }
  __syncthreads();
  float inv = __uint_as_float(scr[0]);
  unsigned bad = scr[1];
  unsigned last = __builtin_amdgcn_readfirstlane(scr[2]);
  STAMP(7);

  auto ors = __builtin_amdgcn_make_buffer_rsrc((void*)pout, 0, NR * MSH * NSH, 0x00020000);
  const unsigned ooff = (shard * MSH + panel * BM + wr * 64 + (lane & 15)) * NSH + colt * BN + wc * 64 + (lane >> 4) * 16;
  const f32x2 magic = {12582912.0f, 12582912.0f};
#pragma unroll 1
  for (;;) {
    const f32x2 inv2 = {inv, inv};
#pragma unroll
    for (int m = 0; m < 4; ++m) {
      unsigned d[4];
#pragma unroll
      for (int j = 0; j < 4; ++j) {
        s16x2 h[2];
#pragma unroll
        for (int r = 0; r < 2; ++r) {
          const f32x2 y = {acc[m][j][2 * r], acc[m][j][2 * r + 1]};
          const f32x2 q = __builtin_elementwise_fma(y, inv2, magic);
          const unsigned pk = __builtin_amdgcn_perm(__float_as_uint(q[1]), __float_as_uint(q[0]), 0x05040100u);
          s16x2 c = __builtin_bit_cast(s16x2, pk);
          c = __builtin_elementwise_min(c, (s16x2){127, 127});
          h[r] = __builtin_elementwise_max(c, (s16x2){-127, -127});
        }
        d[j] = __builtin_amdgcn_perm(__builtin_bit_cast(unsigned, h[1]), __builtin_bit_cast(unsigned, h[0]), 0x06040200u);
      }
      auto s0 = __builtin_amdgcn_permlane32_swap(d[0], d[2], false, false);
      auto s1 = __builtin_amdgcn_permlane32_swap(d[1], d[3], false, false);
      auto u0 = __builtin_amdgcn_permlane16_swap(s0[0], s1[0], false, false);
      auto u1 = __builtin_amdgcn_permlane16_swap(s0[1], s1[1], false, false);
      u32x4 o = {u0[0], u0[1], u1[0], u1[1]};
      if (bad) o = (u32x4){0x80808080u, 0x80808080u, 0x80808080u, 0x80808080u};
      __builtin_amdgcn_raw_buffer_store_b128(o, ors, ooff + m * 16 * NSH, 0, 17);
    }
    if (last) break;
    if (wave == 0) {
      bool timed_out;
      wait_granules<__HIP_MEMORY_SCOPE_SYSTEM>(g2, gv, 0, seq, tmo, lane, timed_out);
      const unsigned fin = granule_max(gv, seq);
      const float amax = __uint_as_float(fin);
      if (lane == 0) { scr[12] = __float_as_uint(amax > 0.f ? 127.0f / amax : 0.f); scr[13] = timed_out; scr[14] = fin != cur || timed_out; }
    }
    __syncthreads();
    STAMP(5);
    if (!__builtin_amdgcn_readfirstlane(scr[14])) break;
    inv = __uint_as_float(scr[12]);
    bad = scr[13];
    last = 1;
    asm volatile("s_waitcnt vmcnt(0)" ::: "memory");
    STAMP(10);
  }
  STAMP(6);
}

}

extern "C" void kernel_launch(int rank, int nranks, ncclComm_t comm,
                              hipStream_t stream,
                              void* const* d_in, const int* in_sizes, int n_in,
                              void* d_out, int out_size,
                              const int* in_dtypes, int out_dtype,
                              void* const* d_out_all,
                              void* const* d_ws_all, size_t ws_bytes) {
  using namespace agq;
  if (nranks != NR || (unsigned)rank >= (unsigned)NR || n_in != 2 || ws_bytes < WS_NEED ||
      in_sizes[0] != MSH * KD || in_sizes[1] != KD * NSH || out_size != NR * MSH * NSH) {
    fprintf(stderr, "ag_gemm_q8: unexpected problem shape\n");
    return;
  }
  static bool configured = false;
  if (!configured) {
    HIP_CHECK(hipFuncSetAttribute((const void*)ag_gemm_q8, hipFuncAttributeMaxDynamicSharedMemorySize, LDS_BYTES));
    configured = true;
  }
  Params p;
  p.x = (const char*)d_in[0];
  p.w = (const char*)d_in[1];
  p.out = (char*)d_out;
  for (int i = 0; i < NR; ++i) p.ws[i] = (char*)d_ws_all[i];
  p.myws = (char*)d_ws_all[rank];
  p.rank = rank;
  hipLaunchKernelGGL(ag_gemm_q8, dim3(NWG), dim3(NTHR), LDS_BYTES, stream, p);
  HIP_CHECK(hipGetLastError());
}
```

```cpp
#include <hip/hip_runtime.h>
#include <rccl/rccl.h>
#include <cstdio>
#include <cstdint>

#define HIP_CHECK(expr) do { hipError_t _e = (expr); if (_e != hipSuccess) { \
    fprintf(stderr, "HIP error %s at %s:%d\n", hipGetErrorString(_e), __FILE__, __LINE__); \
    return; } } while (0)

namespace agq {

constexpr int NR = 8;
constexpr int MSH = 512, KD = 4096, NSH = 2048;
constexpr int BM = 128, BN = 256, BK = 64, NT = KD / BK;
constexpr int NWG = 256, NTHR = 512;
constexpr int A_TILE = BM * BK * 2;
constexpr int W_TILE = BK * BN * 2;
constexpr int STAGE = A_TILE + W_TILE;
constexpr int NSTAGE = 3;
constexpr int LDS_SCR = NSTAGE * STAGE;
constexpr int LDS_BYTES = LDS_SCR + 512;
constexpr int NEARLY = 2;
constexpr int SPEC_T = 8;
constexpr size_t WS_CTR = 0;
constexpr size_t WS_TMO = 32768;
constexpr size_t WS_FLAG = 65536;
constexpr size_t WS_G1 = WS_FLAG + (size_t)NR * NWG * 64;
constexpr size_t WS_G2 = WS_G1 + (size_t)NWG * 128;
constexpr size_t WS_DBG = WS_G2 + (size_t)NR * 8 * 128;
constexpr size_t WS_X = 1u << 20;
constexpr unsigned XSTRIDE = 4 * A_TILE;
constexpr size_t WS_NEED = WS_X + (size_t)NT * XSTRIDE;
static_assert(WS_DBG + NWG * 128 <= WS_X, "layout");
constexpr unsigned long long DEADLINE_TICKS = 50000000ull;

typedef short s16x4 __attribute__((ext_vector_type(4)));
typedef short s16x8 __attribute__((ext_vector_type(8)));
typedef float f32x4 __attribute__((ext_vector_type(4)));
typedef float f32x2 __attribute__((ext_vector_type(2)));
typedef short s16x2 __attribute__((ext_vector_type(2)));
typedef unsigned u32x4 __attribute__((ext_vector_type(4)));
#define LAS __attribute__((address_space(3)))
#define GAS __attribute__((address_space(1)))

typedef GAS char gchar;
typedef GAS unsigned gu32;
typedef GAS unsigned long long gu64;

struct Params {
  const char* x;
  const char* w;
  char* out;
  char* myws;
  char* ws[NR];
  int rank;
};

#define STAMPW(i) do {} while (0)
#define STAMP(i) do {} while (0)

template <int IMM>
__device__ __forceinline__ void glds16(const gchar* base, unsigned voff, unsigned m0v) {
  asm volatile("s_mov_b32 m0, %2\n\ts_nop 0\n\tglobal_load_lds_dwordx4 %0, %1 offset:%3"
               :: "v"(voff), "s"(base), "s"(m0v), "n"(IMM) : "memory");
}
template <int CTRL>
__device__ __forceinline__ unsigned dpp_max(unsigned v) {
  return max(v, (unsigned)__builtin_amdgcn_update_dpp(0, (int)v, CTRL, 0xf, 0xf, true));
}
__device__ __forceinline__ unsigned wave_max(unsigned v) {
  v = dpp_max<0xB1>(v);
  v = dpp_max<0x4E>(v);
  v = dpp_max<0x141>(v);
  v = dpp_max<0x140>(v);
  const unsigned a = __builtin_amdgcn_readlane(v, 0), b = __builtin_amdgcn_readlane(v, 16);
  const unsigned c = __builtin_amdgcn_readlane(v, 32), d = __builtin_amdgcn_readlane(v, 48);
  return max(max(a, b), max(c, d));
}
__device__ __forceinline__ bool give_up(unsigned seq, unsigned long long t0, gu32* tmo, int lane) {
  const unsigned tv = __hip_atomic_load(tmo, __ATOMIC_RELAXED, __HIP_MEMORY_SCOPE_AGENT);
  if (__builtin_amdgcn_readfirstlane(tv) == seq || __builtin_amdgcn_s_memrealtime() - t0 > DEADLINE_TICKS) {
    if (lane == 0) __hip_atomic_store(tmo, seq, __ATOMIC_RELAXED, __HIP_MEMORY_SCOPE_AGENT);
    return true;
  }
  return false;
}
template <int SCOPE>
__device__ __forceinline__ int wait_granules(const gu64* g, unsigned long long& v, int allow, unsigned seq, gu32* tmo, int lane,
                                             bool& timed_out) {
  const unsigned long long t0 = __builtin_amdgcn_s_memrealtime();
  for (unsigned n = 1;; ++n) {
    const unsigned tv = __hip_atomic_load(tmo, __ATOMIC_RELAXED, __HIP_MEMORY_SCOPE_AGENT);
    if ((unsigned)(v >> 32) != seq) v = __hip_atomic_load(g, __ATOMIC_RELAXED, SCOPE);
    const int miss = __builtin_popcountll(__ballot((unsigned)(v >> 32) != seq));
    timed_out = __builtin_amdgcn_readfirstlane(tv) == seq;
    if (miss <= allow || timed_out) return miss;
    if (miss > allow + 12) __builtin_amdgcn_s_sleep(4);
    if ((n & 15) == 0 && give_up(seq, t0, tmo, lane)) { timed_out = true; return miss; }
  }
}
__device__ __forceinline__ unsigned granule_max(unsigned long long v, unsigned seq) {
  return wave_max((unsigned)(v >> 32) == seq ? (unsigned)v : 0u);
}

__global__ void __launch_bounds__(NTHR, 2) ag_gemm_q8(Params p) {
  extern __shared__ __attribute__((aligned(16))) char smem_raw[];
  LAS char* smem = (LAS char*)smem_raw;
  const int tid = threadIdx.x, lane = tid & 63;
  const int wave = __builtin_amdgcn_readfirstlane(tid >> 6);
  const int bid = blockIdx.x;
  gchar* myws = (gchar*)p.myws;
  gchar* sws = (gchar*)p.ws[bid & 7];
  const gchar* px = (const gchar*)p.x;
  const gchar* pw = (const gchar*)p.w;
  gchar* pout = (gchar*)p.out;
  int rank = p.rank;
  gchar* wsw = (gchar*)p.ws[wave];
  asm volatile("" : "+s"(wsw));
  asm volatile("" : "+s"(myws), "+s"(sws), "+s"(px), "+s"(pw), "+s"(pout), "+s"(rank));
  STAMP(0);

  const int shard = bid & 7, idx = bid >> 3, panel = idx & 3, colt = idx >> 2;
  const int wr = wave >> 2, wc = wave & 3;
  const unsigned lds0 = (unsigned)(size_t)smem;
  unsigned wvoff;
  {
    const int r = lane >> 3;
    const int f = ((r >> 1) & 1) * 2 + (wave & 1) * 4;
    wvoff = (wave * 8 + r) * (NSH * 2) + (((lane & 7) ^ f) * 16);
  }
  const gchar* wbase = pw + colt * (BN * 2);
  const int arb = (wave + colt) & 7;
  const unsigned avoff = arb * 2048 + lane * 16;
  const gchar* abase = sws + WS_X + (size_t)panel * A_TILE;
  const unsigned wm0 = lds0 + wave * 1024, am0 = lds0 + W_TILE + arb * 2048;
#define ISSUE_W(wp, so, i) glds16<(i) * 128>(wp, wvoff, wm0 + (so) + (i) * (8192 - 128))
#define ISSUE_A(ap, so, h) glds16<(h) * 1024>(ap, avoff, am0 + (so))
  LAS unsigned* scr = (LAS unsigned*)(smem + LDS_SCR);

  const int cpan = bid & 3, crb = (bid >> 2) & 7, ckg = bid >> 5;
  const bool early = ckg < NEARLY;
  gu32* const ctr = (gu32*)(myws + WS_CTR + (size_t)bid * 128);
  gu32* const myflag = (gu32*)(wsw + WS_FLAG + ((size_t)rank * NWG + bid) * 64);
  const gchar* xsrc = px + (size_t)(cpan * BM + crb * 16 + (lane & 15)) * (KD * 2) + (ckg * 8 + wave) * (BK * 2) + (lane >> 4) * 16;
  const auto xrs = __builtin_amdgcn_make_buffer_rsrc((void*)(myws + WS_X), 0, NT * XSTRIDE, 0x00020000);
  const unsigned xoff = (ckg * 8 + wave) * XSTRIDE + cpan * A_TILE + crb * 2048 + lane * 16;
  unsigned seq_v;
  u32x4 v0, v1;
  if (early) {
    asm volatile("global_load_dword %0, %3, %4 sc1\n\tglobal_load_dwordx4 %1, %5, off\n\tglobal_load_dwordx4 %2, %5, off offset:64\n\t"
                 "s_waitcnt vmcnt(0)"
                 : "=&v"(seq_v), "=&v"(v0), "=&v"(v1) : "v"(0), "s"(ctr), "v"(xsrc) : "memory");
    __builtin_amdgcn_raw_buffer_store_b128(v0, xrs, xoff, 0, 17);
    __builtin_amdgcn_raw_buffer_store_b128(v1, xrs, xoff + 1024, 0, 17);
    asm volatile("s_waitcnt vmcnt(0)" ::: "memory");
    __builtin_amdgcn_s_barrier();
    asm volatile("" ::: "memory");
    if (lane == 0) __hip_atomic_store(myflag, __builtin_amdgcn_readfirstlane(seq_v) + 1u, __ATOMIC_RELAXED, __HIP_MEMORY_SCOPE_SYSTEM);
    STAMP(1);
    ISSUE_W(wbase, 0, 0); ISSUE_W(wbase, 0, 1); ISSUE_W(wbase, 0, 2); ISSUE_W(wbase, 0, 3);
  } else {
    asm volatile("global_load_dword %0, %1, %2 sc1" : "=&v"(seq_v) : "v"(0), "s"(ctr) : "memory");
    ISSUE_W(wbase, 0, 0); ISSUE_W(wbase, 0, 1); ISSUE_W(wbase, 0, 2); ISSUE_W(wbase, 0, 3);
    asm volatile("s_waitcnt vmcnt(4)" : "+v"(seq_v) :: "memory");
  }
  const unsigned seq = __builtin_amdgcn_readfirstlane(seq_v) + 1u;

  gu32* tmo = (gu32*)(myws + WS_TMO);
  const gu32* fl = (const gu32*)(myws + WS_FLAG + ((size_t)shard * NWG + lane * 4 + panel) * 64);
  auto wait_flags = [&](int nlanes) {
    const unsigned long long t0 = __builtin_amdgcn_s_memrealtime();
    unsigned fv = lane < nlanes ? 0u : seq;
    for (unsigned n = 1;; ++n) {
      if (fv < seq) fv = __hip_atomic_load(fl, __ATOMIC_RELAXED, __HIP_MEMORY_SCOPE_SYSTEM);
      if (__all(fv >= seq)) break;
      if ((n & 15) == 0 && give_up(seq, t0, tmo, lane)) break;
    }
  };
  gchar* peer_ws = (gchar*)p.ws[lane & 7];
  if (wave == 0) { wait_flags(8 * NEARLY); asm volatile("" : "+v"(peer_ws)); STAMP(8); }
  asm volatile("" ::: "memory");
  __builtin_amdgcn_s_barrier();
  asm volatile("" ::: "memory");
  STAMP(2);
  if (tid == 64 * 2) __hip_atomic_store(ctr, seq, __ATOMIC_RELAXED, __HIP_MEMORY_SCOPE_AGENT);
  ISSUE_A(abase, 0, 0); ISSUE_A(abase, 0, 1);
  __builtin_amdgcn_s_barrier();
  {
    const gchar* w1 = wbase + BK * NSH * 2;
    ISSUE_W(w1, STAGE, 0); ISSUE_W(w1, STAGE, 1); ISSUE_W(w1, STAGE, 2); ISSUE_W(w1, STAGE, 3);
  }
  ISSUE_A(abase + XSTRIDE, STAGE, 0); ISSUE_A(abase + XSTRIDE, STAGE, 1);

  unsigned wadr[4];
  {
    const int g = lane >> 4, q = (lane & 15) >> 2, pp = lane & 3;
    const int f = (q >> 1) * 2 + (g & 1) * 4;
#pragma unroll
    for (int j = 0; j < 4; ++j) wadr[j] = wc * 8192 + g * 1024 + q * 128 + (((2 * j + (pp >> 1)) ^ f) * 16) + (pp & 1) * 8;
  }
  const unsigned aadr = W_TILE + wr * 8192 + lane * 16;

  f32x4 acc[4][4];
#pragma unroll
  for (int m = 0; m < 4; ++m)
#pragma unroll
    for (int j = 0; j < 4; ++j) acc[m][j] = (f32x4){0.f, 0.f, 0.f, 0.f};

  s16x8 af[2][4], wf[2][4];
  auto read_w = [&](int so, int ks) {
#pragma unroll
    for (int j = 0; j < 4; ++j) {
      const s16x4 lo = __builtin_amdgcn_ds_read_tr16_b64_v4i16((LAS s16x4*)(smem + so + wadr[j] + ks * 4096));
      const s16x4 hi = __builtin_amdgcn_ds_read_tr16_b64_v4i16((LAS s16x4*)(smem + so + wadr[j] + ks * 4096 + 512));
      wf[ks][j] = __builtin_shufflevector(lo, hi, 0, 1, 2, 3, 4, 5, 6, 7);
    }
  };
  auto read_a = [&](int so, int ks) {
#pragma unroll
    for (int m = 0; m < 4; ++m) af[ks][m] = *(const LAS s16x8*)(smem + so + aadr + m * 2048 + ks * 1024);
  };
  auto mma = [&](int ks, int m) {
#pragma unroll
    for (int j = 0; j < 4; ++j)
      acc[m][j] = __builtin_amdgcn_mfma_f32_16x16x32_bf16(wf[ks][j], af[ks][m], acc[m][j], 0, 0, 0);
    __builtin_amdgcn_sched_barrier(0);
  };
#define TILE_SYNC(n) do { asm volatile("s_waitcnt vmcnt(%0) lgkmcnt(0)" :: "n"(n) : "memory"); __builtin_amdgcn_s_barrier(); asm volatile("" ::: "memory"); } while (0)

  TILE_SYNC(6);
  STAMP(11);
  int so = 0;
  const int tl = early ? -8 : 3 * (ckg - NEARLY) + 1;
  for (int t = 0; t < NT - 2; ++t) {
    const int dt = t - tl;
    if (dt == 4) {
      if (lane == 0) __hip_atomic_store(myflag, seq, __ATOMIC_RELAXED, __HIP_MEMORY_SCOPE_SYSTEM);
    }
    if (wave == 0 && t < 32) {
      if ((t & 15) == 12) {
        asm volatile("s_mov_b32 m0, %1\n\ts_nop 0\n\tglobal_load_lds_dword %0, off sc0 sc1" :: "v"(fl), "s"(lds0 + LDS_SCR + 256) : "memory");
      } else if ((t & 15) == 13) {
        const unsigned pv = scr[64 + lane];
        const int nl = t < 16 ? 32 : 64;
        if (!__all(pv >= seq || lane >= nl)) wait_flags(nl);
      }
    }
    const gchar* ap = abase + (size_t)(t + 2) * XSTRIDE;
    const gchar* wp = wbase + (size_t)(t + 2) * (BK * NSH * 2);
    const int sn = so == 0 ? 2 * STAGE : so - STAGE;
    read_w(so, 0); read_a(so, 0);
    ISSUE_W(wp, sn, 0);
    read_w(so, 1); read_a(so, 1);
    __builtin_amdgcn_sched_barrier(0);
    mma(0, 0); ISSUE_W(wp, sn, 1); __builtin_amdgcn_sched_barrier(0);
    mma(0, 1); ISSUE_W(wp, sn, 2); __builtin_amdgcn_sched_barrier(0);
    mma(0, 2); ISSUE_W(wp, sn, 3); __builtin_amdgcn_sched_barrier(0);
    mma(0, 3); ISSUE_A(ap, sn, 0); __builtin_amdgcn_sched_barrier(0);
    mma(1, 0); ISSUE_A(ap, sn, 1); __builtin_amdgcn_sched_barrier(0);
    mma(1, 1); mma(1, 2); mma(1, 3);
    so = so == 2 * STAGE ? 0 : so + STAGE;
    if (dt == 0) {
      asm volatile("global_load_dwordx4 v[240:243], %0, off\n\tglobal_load_dwordx4 v[244:247], %0, off offset:64" :: "v"(xsrc)
                   : "memory", "v240", "v241", "v242", "v243", "v244", "v245", "v246", "v247");
      TILE_SYNC(8);
    } else if (dt == 2) {
      asm volatile("global_store_dwordx4 %0, v[240:243], %1 sc0 sc1\n\tglobal_store_dwordx4 %0, v[244:247], %1 offset:1024 sc0 sc1\n\ts_nop 1"
                   :: "v"(xoff), "s"(myws + WS_X) : "memory");
      TILE_SYNC(8);
    } else {
      TILE_SYNC(6);
    }
  }
  read_w(so, 0); read_a(so, 0); read_w(so, 1); read_a(so, 1);
  __builtin_amdgcn_sched_barrier(0);
  mma(0, 0); mma(0, 1); mma(0, 2); mma(0, 3); mma(1, 0); mma(1, 1); mma(1, 2); mma(1, 3);
  so = so == 2 * STAGE ? 0 : so + STAGE;
  TILE_SYNC(1);
  read_w(so, 0); read_a(so, 0); read_w(so, 1);
  __builtin_amdgcn_sched_barrier(0);
  mma(0, 0); mma(0, 1); mma(0, 2); mma(0, 3);
  TILE_SYNC(0);
  read_a(so, 1);
  __builtin_amdgcn_sched_barrier(0);
  mma(1, 0); mma(1, 1); mma(1, 2); mma(1, 3);
  STAMP(3);

  float mx = 0.f;
#pragma unroll
  for (int m = 0; m < 4; ++m)
#pragma unroll
    for (int j = 0; j < 4; ++j)
#pragma unroll
      for (int r = 0; r < 4; ++r) mx = fmaxf(mx, fabsf(acc[m][j][r]));
  const unsigned wmx = wave_max(__float_as_uint(mx));
  if (lane == 0) scr[4 + wave] = wmx;
  __syncthreads();
  const gu64* g2 = (const gu64*)(myws + WS_G2 + (size_t)lane * 128);
  unsigned long long gv = 0;
  unsigned cur = 0;
  if (wave == 0) {
    const u32x4 ra = *(const LAS u32x4*)(scr + 4), rb = *(const LAS u32x4*)(scr + 8);
    const unsigned gm = max(max(max(ra[0], ra[1]), max(ra[2], ra[3])), max(max(rb[0], rb[1]), max(rb[2], rb[3])));
    gu64* g1 = (gu64*)(myws + WS_G1 + (size_t)shard * 32 * 128);
    if (lane == 0) __hip_atomic_store(g1 + idx * 16, ((unsigned long long)seq << 32) | gm, __ATOMIC_RELAXED, __HIP_MEMORY_SCOPE_AGENT);
    STAMP(4);
    bool timed_out;
    if (idx == 0) {
      unsigned long long v = 0;
      wait_granules<__HIP_MEMORY_SCOPE_AGENT>(g1 + (lane & 31) * 16, v, 0, seq, tmo, lane, timed_out);
      const unsigned gmax = granule_max(v, seq);
      if (lane < NR && !timed_out)
        __hip_atomic_store((gu64*)(peer_ws + WS_G2 + ((size_t)rank * 8 + shard) * 128),
                           ((unsigned long long)seq << 32) | gmax, __ATOMIC_RELAXED, __HIP_MEMORY_SCOPE_SYSTEM);
      STAMP(9);
    }
    const int miss = wait_granules<__HIP_MEMORY_SCOPE_SYSTEM>(g2, gv, SPEC_T, seq, tmo, lane, timed_out);
    cur = max(granule_max(gv, seq), gm);
    const float amax = __uint_as_float(cur);
    if (lane == 0) { scr[0] = __float_as_uint(amax > 0.f ? 127.0f / amax : 0.f); scr[1] = timed_out; scr[2] = miss == 0 || timed_out; }
  }
  __syncthreads();
  float inv = __uint_as_float(scr[0]);
  unsigned bad = scr[1];
  unsigned last = __builtin_amdgcn_readfirstlane(scr[2]);
  STAMP(7);

  auto ors = __builtin_amdgcn_make_buffer_rsrc((void*)pout, 0, NR * MSH * NSH, 0x00020000);
  const unsigned ooff = (shard * MSH + panel * BM + wr * 64 + (lane & 15)) * NSH + colt * BN + wc * 64 + (lane >> 4) * 16;
  const f32x2 magic = {12582912.0f, 12582912.0f};
#pragma unroll 1
  for (;;) {
    const f32x2 inv2 = {inv, inv};
#pragma unroll
    for (int m = 0; m < 4; ++m) {
      unsigned d[4];
#pragma unroll
      for (int j = 0; j < 4; ++j) {
        s16x2 h[2];
#pragma unroll
        for (int r = 0; r < 2; ++r) {
          const f32x2 y = {acc[m][j][2 * r], acc[m][j][2 * r + 1]};
          const f32x2 q = __builtin_elementwise_fma(y, inv2, magic);
          const unsigned pk = __builtin_amdgcn_perm(__float_as_uint(q[1]), __float_as_uint(q[0]), 0x05040100u);
          s16x2 c = __builtin_bit_cast(s16x2, pk);
          c = __builtin_elementwise_min(c, (s16x2){127, 127});
          h[r] = __builtin_elementwise_max(c, (s16x2){-127, -127});
        }
        d[j] = __builtin_amdgcn_perm(__builtin_bit_cast(unsigned, h[1]), __builtin_bit_cast(unsigned, h[0]), 0x06040200u);
      }
      auto s0 = __builtin_amdgcn_permlane32_swap(d[0], d[2], false, false);
      auto s1 = __builtin_amdgcn_permlane32_swap(d[1], d[3], false, false);
      auto u0 = __builtin_amdgcn_permlane16_swap(s0[0], s1[0], false, false);
      auto u1 = __builtin_amdgcn_permlane16_swap(s0[1], s1[1], false, false);
      u32x4 o = {u0[0], u0[1], u1[0], u1[1]};
      if (bad) o = (u32x4){0x80808080u, 0x80808080u, 0x80808080u, 0x80808080u};
      __builtin_amdgcn_raw_buffer_store_b128(o, ors, ooff + m * 16 * NSH, 0, 17);
    }
    if (last) break;
    if (wave == 0) {
      bool timed_out;
      wait_granules<__HIP_MEMORY_SCOPE_SYSTEM>(g2, gv, 0, seq, tmo, lane, timed_out);
      const unsigned fin = granule_max(gv, seq);
      const float amax = __uint_as_float(fin);
      if (lane == 0) { scr[12] = __float_as_uint(amax > 0.f ? 127.0f / amax : 0.f); scr[13] = timed_out; scr[14] = fin != cur || timed_out; }
    }
    __syncthreads();
    STAMP(5);
    if (!__builtin_amdgcn_readfirstlane(scr[14])) break;
    inv = __uint_as_float(scr[12]);
    bad = scr[13];
    last = 1;
    asm volatile("s_waitcnt vmcnt(0)" ::: "memory");
    STAMP(10);
  }
  STAMP(6);
}

}

extern "C" void kernel_launch(int rank, int nranks, ncclComm_t comm,
                              hipStream_t stream,
                              void* const* d_in, const int* in_sizes, int n_in,
                              void* d_out, int out_size,
                              const int* in_dtypes, int out_dtype,
                              void* const* d_out_all,
                              void* const* d_ws_all, size_t ws_bytes) {
  using namespace agq;
  if (nranks != NR || (unsigned)rank >= (unsigned)NR || n_in != 2 || ws_bytes < WS_NEED ||
      in_sizes[0] != MSH * KD || in_sizes[1] != KD * NSH || out_size != NR * MSH * NSH) {
    fprintf(stderr, "ag_gemm_q8: unexpected problem shape\n");
    return;
  }
  static bool configured = false;
  if (!configured) {
    int dev = 0, cus = 0;
    HIP_CHECK(hipGetDevice(&dev));
    HIP_CHECK(hipDeviceGetAttribute(&cus, hipDeviceAttributeMultiprocessorCount, dev));
    if (cus < NWG) fprintf(stderr, "ag_gemm_q8: needs %d CUs, found %d\n", NWG, cus);
    HIP_CHECK(hipFuncSetAttribute((const void*)ag_gemm_q8, hipFuncAttributeMaxDynamicSharedMemorySize, LDS_BYTES));
    configured = true;
  }
  Params p;
  p.x = (const char*)d_in[0];
  p.w = (const char*)d_in[1];
  p.out = (char*)d_out;
  for (int i = 0; i < NR; ++i) p.ws[i] = (char*)d_ws_all[i];
  p.myws = (char*)d_ws_all[rank];
  p.rank = rank;
  hipLaunchKernelGGL(ag_gemm_q8, dim3(NWG), dim3(NTHR), LDS_BYTES, stream, p);
  HIP_CHECK(hipGetLastError());
}
```

```cpp
#include <hip/hip_runtime.h>
#include <rccl/rccl.h>
#include <cstdio>
#include <cstdint>

#define HIP_CHECK(expr) do { hipError_t _e = (expr); if (_e != hipSuccess) { \
    fprintf(stderr, "HIP error %s at %s:%d\n", hipGetErrorString(_e), __FILE__, __LINE__); \
    return; } } while (0)

namespace agq {

constexpr int NR = 8;
constexpr int MSH = 512, KD = 4096, NSH = 2048;
constexpr int BM = 128, BN = 256, BK = 64, NT = KD / BK;
constexpr int NWG = 256, NTHR = 512;
constexpr int A_TILE = BM * BK * 2;
constexpr int W_TILE = BK * BN * 2;
constexpr int STAGE = A_TILE + W_TILE;
constexpr int NSTAGE = 3;
constexpr int LDS_SCR = NSTAGE * STAGE;
constexpr int LDS_BYTES = LDS_SCR + 512;
constexpr int NEARLY = 2;
constexpr int SPEC_T = 8;
constexpr size_t WS_CTR = 0;
constexpr size_t WS_TMO = 32768;
constexpr size_t WS_FLAG = 65536;
constexpr size_t WS_G1 = WS_FLAG + (size_t)NR * NWG * 64;
constexpr size_t WS_G2 = WS_G1 + (size_t)NWG * 128;
constexpr size_t WS_DBG = WS_G2 + (size_t)NR * 8 * 128;
constexpr size_t WS_X = 1u << 20;
constexpr unsigned XSTRIDE = 4 * A_TILE;
constexpr size_t WS_NEED = WS_X + (size_t)NT * XSTRIDE;
static_assert(WS_DBG + NWG * 128 <= WS_X, "layout");
constexpr unsigned long long DEADLINE_TICKS = 100000000ull;

typedef short s16x4 __attribute__((ext_vector_type(4)));
typedef short s16x8 __attribute__((ext_vector_type(8)));
typedef float f32x4 __attribute__((ext_vector_type(4)));
typedef float f32x2 __attribute__((ext_vector_type(2)));
typedef short s16x2 __attribute__((ext_vector_type(2)));
typedef unsigned u32x4 __attribute__((ext_vector_type(4)));
#define LAS __attribute__((address_space(3)))
#define GAS __attribute__((address_space(1)))

typedef GAS char gchar;
typedef GAS unsigned gu32;
typedef GAS unsigned long long gu64;

struct Params {
  const char* x;
  const char* w;
  char* out;
  char* myws;
  char* ws[NR];
  int rank;
};

#define STAMPW(i) do {} while (0)
#define STAMP(i) do {} while (0)

template <int IMM>
__device__ __forceinline__ void glds16(const gchar* base, unsigned voff, unsigned m0v) {
  asm volatile("s_mov_b32 m0, %2\n\ts_nop 0\n\tglobal_load_lds_dwordx4 %0, %1 offset:%3"
               :: "v"(voff), "s"(base), "s"(m0v), "n"(IMM) : "memory");
}
template <int CTRL>
__device__ __forceinline__ unsigned dpp_max(unsigned v) {
  return max(v, (unsigned)__builtin_amdgcn_update_dpp(0, (int)v, CTRL, 0xf, 0xf, true));
}
__device__ __forceinline__ unsigned wave_max(unsigned v) {
  v = dpp_max<0xB1>(v);
  v = dpp_max<0x4E>(v);
  v = dpp_max<0x141>(v);
  v = dpp_max<0x140>(v);
  const unsigned a = __builtin_amdgcn_readlane(v, 0), b = __builtin_amdgcn_readlane(v, 16);
  const unsigned c = __builtin_amdgcn_readlane(v, 32), d = __builtin_amdgcn_readlane(v, 48);
  return max(max(a, b), max(c, d));
}
__device__ __forceinline__ bool give_up(unsigned seq, unsigned long long t0, gu32* tmo, int lane) {
  const unsigned tv = __hip_atomic_load(tmo, __ATOMIC_RELAXED, __HIP_MEMORY_SCOPE_AGENT);
  if (__builtin_amdgcn_readfirstlane(tv) == seq || __builtin_amdgcn_s_memrealtime() - t0 > DEADLINE_TICKS) {
    if (lane == 0) __hip_atomic_store(tmo, seq, __ATOMIC_RELAXED, __HIP_MEMORY_SCOPE_AGENT);
    return true;
  }
  return false;
}
template <int SCOPE>
__device__ __forceinline__ int wait_granules(const gu64* g, unsigned long long& v, int allow, unsigned seq, gu32* tmo, int lane,
                                             bool& timed_out) {
  const unsigned long long t0 = __builtin_amdgcn_s_memrealtime();
  for (unsigned n = 1;; ++n) {
    const unsigned tv = __hip_atomic_load(tmo, __ATOMIC_RELAXED, __HIP_MEMORY_SCOPE_AGENT);
    if ((unsigned)(v >> 32) != seq) v = __hip_atomic_load(g, __ATOMIC_RELAXED, SCOPE);
    const int miss = __builtin_popcountll(__ballot((unsigned)(v >> 32) != seq));
    timed_out = __builtin_amdgcn_readfirstlane(tv) == seq;
    if (miss <= allow || timed_out) return miss;
    if (miss > allow + 12) __builtin_amdgcn_s_sleep(4);
    if ((n & 15) == 0 && give_up(seq, t0, tmo, lane)) { timed_out = true; return miss; }
  }
}
__device__ __forceinline__ unsigned granule_max(unsigned long long v, unsigned seq) {
  return wave_max((unsigned)(v >> 32) == seq ? (unsigned)v : 0u);
}

__global__ void __launch_bounds__(NTHR, 2) ag_gemm_q8(Params p) {
  extern __shared__ __attribute__((aligned(16))) char smem_raw[];
  LAS char* smem = (LAS char*)smem_raw;
  const int tid = threadIdx.x;
  int lane = tid & 63;
  const int wave = __builtin_amdgcn_readfirstlane(tid >> 6);
  int bid = blockIdx.x;
  const unsigned lds0 = (unsigned)(size_t)smem;
  const int ckg = bid >> 5;
  const bool early = ckg < NEARLY;
  gchar* myws = (gchar*)p.myws;
  gchar* sws = (gchar*)p.ws[bid & 7];
  const gchar* px = (const gchar*)p.x;
  const gchar* pw = (const gchar*)p.w;
  gchar* pout = (gchar*)p.out;
  int rank = p.rank;
  gchar* wsw = (gchar*)p.ws[wave];
  asm volatile("" : "+s"(myws), "+s"(sws), "+s"(wsw), "+s"(px), "+s"(pw), "+s"(pout), "+s"(rank));
  STAMP(0);

  const int cpan = bid & 3, crb = (bid >> 2) & 7;
  gu32* const ctr = (gu32*)(myws + WS_CTR + (size_t)bid * 128);
  const gchar* xsrc = px + (size_t)(cpan * BM + crb * 16 + (lane & 15)) * (KD * 2) + (ckg * 8 + wave) * (BK * 2) + (lane >> 4) * 16;
  if (early) {
    asm volatile("global_load_dwordx4 v[240:243], %0, off\n\tglobal_load_dwordx4 v[244:247], %0, off offset:64" :: "v"(xsrc)
                 : "memory", "v240", "v241", "v242", "v243", "v244", "v245", "v246", "v247");
  }
  asm volatile("global_load_dword v248, %0, %1 sc1" :: "v"(0), "s"(ctr) : "memory", "v248");
  asm volatile("" : "+s"(bid), "+v"(lane));
  gu32* const myflag = (gu32*)(wsw + WS_FLAG + ((size_t)rank * NWG + bid) * 64);
  const unsigned xoff = ((bid >> 5) * 8 + wave) * XSTRIDE + (bid & 3) * A_TILE + ((bid >> 2) & 7) * 2048 + lane * 16;

  const int shard = bid & 7, idx = bid >> 3, panel = idx & 3, colt = idx >> 2;
  const int wr = wave >> 2, wc = wave & 3;
  unsigned wvoff;
  {
    const int r = lane >> 3;
    const int f = ((r >> 1) & 1) * 2 + (wave & 1) * 4;
    wvoff = (wave * 8 + r) * (NSH * 2) + (((lane & 7) ^ f) * 16);
  }
  const gchar* wbase = pw + colt * (BN * 2);
  const int arb = (wave + colt) & 7;
  const unsigned avoff = arb * 2048 + lane * 16;
  const gchar* abase = sws + WS_X + (size_t)panel * A_TILE;
  const unsigned wm0 = lds0 + wave * 1024, am0 = lds0 + W_TILE + arb * 2048;
#define ISSUE_W(wp, so, i) glds16<(i) * 128>(wp, wvoff, wm0 + (so) + (i) * (8192 - 128))
#define ISSUE_A(ap, so, h) glds16<(h) * 1024>(ap, avoff, am0 + (so))
  LAS unsigned* scr = (LAS unsigned*)(smem + LDS_SCR);
  unsigned seq;
  if (early) {
    asm volatile("s_waitcnt vmcnt(0)\n\tglobal_store_dwordx4 %1, v[240:243], %2 sc0 sc1\n\tglobal_store_dwordx4 %1, v[244:247], %2 offset:1024 sc0 sc1\n\t"
                 "v_readfirstlane_b32 %0, v248\n\ts_waitcnt vmcnt(0)" : "=&s"(seq) : "v"(xoff), "s"(myws + WS_X) : "memory");
    __builtin_amdgcn_s_barrier();
    asm volatile("" ::: "memory");
    seq += 1u;
    if (lane == 0) __hip_atomic_store(myflag, seq, __ATOMIC_RELAXED, __HIP_MEMORY_SCOPE_SYSTEM);
    STAMP(1);
    ISSUE_W(wbase, 0, 0); ISSUE_W(wbase, 0, 1); ISSUE_W(wbase, 0, 2); ISSUE_W(wbase, 0, 3);
  } else {
    ISSUE_W(wbase, 0, 0); ISSUE_W(wbase, 0, 1); ISSUE_W(wbase, 0, 2); ISSUE_W(wbase, 0, 3);
    asm volatile("s_waitcnt vmcnt(4)\n\tv_readfirstlane_b32 %0, v248" : "=s"(seq) :: "memory");
    seq += 1u;
  }

  gu32* tmo = (gu32*)(myws + WS_TMO);
  const gu32* fl = (const gu32*)(myws + WS_FLAG + ((size_t)shard * NWG + lane * 4 + panel) * 64);
  auto wait_flags = [&](int nlanes) {
    const unsigned long long t0 = __builtin_amdgcn_s_memrealtime();
    unsigned fv = lane < nlanes ? 0u : seq;
    for (unsigned n = 1;; ++n) {
      if (fv < seq) fv = __hip_atomic_load(fl, __ATOMIC_RELAXED, __HIP_MEMORY_SCOPE_SYSTEM);
      if (__all(fv >= seq)) break;
      if ((n & 15) == 0 && give_up(seq, t0, tmo, lane)) break;
    }
  };
  gchar* peer_ws = (gchar*)p.ws[lane & 7];
  if (wave == 0) { wait_flags(8 * NEARLY); asm volatile("" : "+v"(peer_ws)); STAMP(8); }
  asm volatile("" ::: "memory");
  __builtin_amdgcn_s_barrier();
  asm volatile("" ::: "memory");
  STAMP(2);
  if (tid == 64 * 2) __hip_atomic_store(ctr, seq, __ATOMIC_RELAXED, __HIP_MEMORY_SCOPE_AGENT);
  ISSUE_A(abase, 0, 0); ISSUE_A(abase, 0, 1);
  __builtin_amdgcn_s_barrier();
  {
    const gchar* w1 = wbase + BK * NSH * 2;
    ISSUE_W(w1, STAGE, 0); ISSUE_W(w1, STAGE, 1); ISSUE_W(w1, STAGE, 2); ISSUE_W(w1, STAGE, 3);
  }
  ISSUE_A(abase + XSTRIDE, STAGE, 0); ISSUE_A(abase + XSTRIDE, STAGE, 1);

  unsigned wadr[4];
  {
    const int g = lane >> 4, q = (lane & 15) >> 2, pp = lane & 3;
    const int f = (q >> 1) * 2 + (g & 1) * 4;
#pragma unroll
    for (int j = 0; j < 4; ++j) wadr[j] = wc * 8192 + g * 1024 + q * 128 + (((2 * j + (pp >> 1)) ^ f) * 16) + (pp & 1) * 8;
  }
  const unsigned aadr = W_TILE + wr * 8192 + lane * 16;

  f32x4 acc[4][4];
#pragma unroll
  for (int m = 0; m < 4; ++m)
#pragma unroll
    for (int j = 0; j < 4; ++j) acc[m][j] = (f32x4){0.f, 0.f, 0.f, 0.f};

  s16x8 af[2][4], wf[2][4];
  auto read_w = [&](int so, int ks) {
#pragma unroll
    for (int j = 0; j < 4; ++j) {
      const s16x4 lo = __builtin_amdgcn_ds_read_tr16_b64_v4i16((LAS s16x4*)(smem + so + wadr[j] + ks * 4096));
      const s16x4 hi = __builtin_amdgcn_ds_read_tr16_b64_v4i16((LAS s16x4*)(smem + so + wadr[j] + ks * 4096 + 512));
      wf[ks][j] = __builtin_shufflevector(lo, hi, 0, 1, 2, 3, 4, 5, 6, 7);
    }
  };
  auto read_a = [&](int so, int ks) {
#pragma unroll
    for (int m = 0; m < 4; ++m) af[ks][m] = *(const LAS s16x8*)(smem + so + aadr + m * 2048 + ks * 1024);
  };
  auto mma = [&](int ks, int m) {
#pragma unroll
    for (int j = 0; j < 4; ++j)
      acc[m][j] = __builtin_amdgcn_mfma_f32_16x16x32_bf16(wf[ks][j], af[ks][m], acc[m][j], 0, 0, 0);
    __builtin_amdgcn_sched_barrier(0);
  };
#define TILE_SYNC(n) do { asm volatile("s_waitcnt vmcnt(%0) lgkmcnt(0)" :: "n"(n) : "memory"); __builtin_amdgcn_s_barrier(); asm volatile("" ::: "memory"); } while (0)

  TILE_SYNC(6);
  STAMP(11);
  int so = 0;
  const int tl = early ? -8 : 3 * (ckg - NEARLY) + 1;
  for (int t = 0; t < NT - 2; ++t) {
    const int dt = t - tl;
    if (dt == 4) {
      if (lane == 0) __hip_atomic_store(myflag, seq, __ATOMIC_RELAXED, __HIP_MEMORY_SCOPE_SYSTEM);
    }
    if (wave == 0 && t < 32) {
      if ((t & 15) == 12) {
        asm volatile("s_mov_b32 m0, %1\n\ts_nop 0\n\tglobal_load_lds_dword %0, off sc0 sc1" :: "v"(fl), "s"(lds0 + LDS_SCR + 256) : "memory");
      } else if ((t & 15) == 13) {
        const unsigned pv = scr[64 + lane];
        const int nl = t < 16 ? 32 : 64;
        if (!__all(pv >= seq || lane >= nl)) wait_flags(nl);
      }
    }
    const gchar* ap = abase + (size_t)(t + 2) * XSTRIDE;
    const gchar* wp = wbase + (size_t)(t + 2) * (BK * NSH * 2);
    const int sn = so == 0 ? 2 * STAGE : so - STAGE;
    read_w(so, 0); read_a(so, 0);
    ISSUE_W(wp, sn, 0);
    read_w(so, 1); read_a(so, 1);
    __builtin_amdgcn_sched_barrier(0);
    mma(0, 0); ISSUE_W(wp, sn, 1); __builtin_amdgcn_sched_barrier(0);
    mma(0, 1); ISSUE_W(wp, sn, 2); __builtin_amdgcn_sched_barrier(0);
    mma(0, 2); ISSUE_W(wp, sn, 3); __builtin_amdgcn_sched_barrier(0);
    mma(0, 3); ISSUE_A(ap, sn, 0); __builtin_amdgcn_sched_barrier(0);
    mma(1, 0); ISSUE_A(ap, sn, 1); __builtin_amdgcn_sched_barrier(0);
    mma(1, 1); mma(1, 2); mma(1, 3);
    so = so == 2 * STAGE ? 0 : so + STAGE;
    if (dt == 0) {
      asm volatile("global_load_dwordx4 v[240:243], %0, off\n\tglobal_load_dwordx4 v[244:247], %0, off offset:64" :: "v"(xsrc)
                   : "memory", "v240", "v241", "v242", "v243", "v244", "v245", "v246", "v247");
      TILE_SYNC(8);
    } else if (dt == 2) {
      asm volatile("global_store_dwordx4 %0, v[240:243], %1 sc0 sc1\n\tglobal_store_dwordx4 %0, v[244:247], %1 offset:1024 sc0 sc1\n\ts_nop 1"
                   :: "v"(xoff), "s"(myws + WS_X) : "memory");
      TILE_SYNC(8);
    } else {
      TILE_SYNC(6);
    }
  }
  read_w(so, 0); read_a(so, 0); read_w(so, 1); read_a(so, 1);
  __builtin_amdgcn_sched_barrier(0);
  mma(0, 0); mma(0, 1); mma(0, 2); mma(0, 3); mma(1, 0); mma(1, 1); mma(1, 2); mma(1, 3);
  so = so == 2 * STAGE ? 0 : so + STAGE;
  TILE_SYNC(1);
  read_w(so, 0); read_a(so, 0); read_w(so, 1);
  __builtin_amdgcn_sched_barrier(0);
  mma(0, 0); mma(0, 1); mma(0, 2); mma(0, 3);
  TILE_SYNC(0);
  read_a(so, 1);
  __builtin_amdgcn_sched_barrier(0);
  mma(1, 0); mma(1, 1); mma(1, 2); mma(1, 3);
  STAMP(3);

  float mx = 0.f;
#pragma unroll
  for (int m = 0; m < 4; ++m)
#pragma unroll
    for (int j = 0; j < 4; ++j)
#pragma unroll
      for (int r = 0; r < 4; ++r) mx = fmaxf(mx, fabsf(acc[m][j][r]));
  const unsigned wmx = wave_max(__float_as_uint(mx));
  if (lane == 0) scr[4 + wave] = wmx;
  __syncthreads();
  const gu64* g2 = (const gu64*)(myws + WS_G2 + (size_t)lane * 128);
  unsigned long long gv = 0;
  unsigned cur = 0;
  if (wave == 0) {
    const u32x4 ra = *(const LAS u32x4*)(scr + 4), rb = *(const LAS u32x4*)(scr + 8);
    const unsigned gm = max(max(max(ra[0], ra[1]), max(ra[2], ra[3])), max(max(rb[0], rb[1]), max(rb[2], rb[3])));
    gu64* g1 = (gu64*)(myws + WS_G1 + (size_t)shard * 32 * 128);
    if (lane == 0) __hip_atomic_store(g1 + idx * 16, ((unsigned long long)seq << 32) | gm, __ATOMIC_RELAXED, __HIP_MEMORY_SCOPE_AGENT);
    STAMP(4);
    bool timed_out;
    if (idx == 0) {
      unsigned long long v = 0;
      wait_granules<__HIP_MEMORY_SCOPE_AGENT>(g1 + (lane & 31) * 16, v, 0, seq, tmo, lane, timed_out);
      const unsigned gmax = granule_max(v, seq);
      if (lane < NR && !timed_out)
        __hip_atomic_store((gu64*)(peer_ws + WS_G2 + ((size_t)rank * 8 + shard) * 128),
                           ((unsigned long long)seq << 32) | gmax, __ATOMIC_RELAXED, __HIP_MEMORY_SCOPE_SYSTEM);
      STAMP(9);
    }
    const int miss = wait_granules<__HIP_MEMORY_SCOPE_SYSTEM>(g2, gv, SPEC_T, seq, tmo, lane, timed_out);
    cur = max(granule_max(gv, seq), gm);
    const float amax = __uint_as_float(cur);
    if (lane == 0) { scr[0] = __float_as_uint(amax > 0.f ? 127.0f / amax : 0.f); scr[1] = timed_out; scr[2] = miss == 0 || timed_out; }
  }
  __syncthreads();
  float inv = __uint_as_float(scr[0]);
  unsigned bad = scr[1];
  unsigned last = __builtin_amdgcn_readfirstlane(scr[2]);
  STAMP(7);

  auto ors = __builtin_amdgcn_make_buffer_rsrc((void*)pout, 0, NR * MSH * NSH, 0x00020000);
  const unsigned ooff = (shard * MSH + panel * BM + wr * 64 + (lane & 15)) * NSH + colt * BN + wc * 64 + (lane >> 4) * 16;
  const f32x2 magic = {12582912.0f, 12582912.0f};
#pragma unroll 1
  for (;;) {
    const f32x2 inv2 = {inv, inv};
#pragma unroll
    for (int m = 0; m < 4; ++m) {
      unsigned d[4];
#pragma unroll
      for (int j = 0; j < 4; ++j) {
        s16x2 h[2];
#pragma unroll
        for (int r = 0; r < 2; ++r) {
          const f32x2 y = {acc[m][j][2 * r], acc[m][j][2 * r + 1]};
          const f32x2 q = __builtin_elementwise_fma(y, inv2, magic);
          const unsigned pk = __builtin_amdgcn_perm(__float_as_uint(q[1]), __float_as_uint(q[0]), 0x05040100u);
          s16x2 c = __builtin_bit_cast(s16x2, pk);
          c = __builtin_elementwise_min(c, (s16x2){127, 127});
          h[r] = __builtin_elementwise_max(c, (s16x2){-127, -127});
        }
        d[j] = __builtin_amdgcn_perm(__builtin_bit_cast(unsigned, h[1]), __builtin_bit_cast(unsigned, h[0]), 0x06040200u);
      }
      auto s0 = __builtin_amdgcn_permlane32_swap(d[0], d[2], false, false);
      auto s1 = __builtin_amdgcn_permlane32_swap(d[1], d[3], false, false);
      auto u0 = __builtin_amdgcn_permlane16_swap(s0[0], s1[0], false, false);
      auto u1 = __builtin_amdgcn_permlane16_swap(s0[1], s1[1], false, false);
      u32x4 o = {u0[0], u0[1], u1[0], u1[1]};
      if (bad) o = (u32x4){0x80808080u, 0x80808080u, 0x80808080u, 0x80808080u};
      __builtin_amdgcn_raw_buffer_store_b128(o, ors, ooff + m * 16 * NSH, 0, 17);
    }
    if (last) break;
    if (wave == 0) {
      bool timed_out;
      wait_granules<__HIP_MEMORY_SCOPE_SYSTEM>(g2, gv, 0, seq, tmo, lane, timed_out);
      const unsigned fin = granule_max(gv, seq);
      const float amax = __uint_as_float(fin);
      if (lane == 0) { scr[12] = __float_as_uint(amax > 0.f ? 127.0f / amax : 0.f); scr[13] = timed_out; scr[14] = fin != cur || timed_out; }
    }
    __syncthreads();
    STAMP(5);
    if (!__builtin_amdgcn_readfirstlane(scr[14])) break;
    inv = __uint_as_float(scr[12]);
    bad = scr[13];
    last = 1;
    asm volatile("s_waitcnt vmcnt(0)" ::: "memory");
    STAMP(10);
  }
  STAMP(6);
}

}

extern "C" void kernel_launch(int rank, int nranks, ncclComm_t comm,
                              hipStream_t stream,
                              void* const* d_in, const int* in_sizes, int n_in,
                              void* d_out, int out_size,
                              const int* in_dtypes, int out_dtype,
                              void* const* d_out_all,
                              void* const* d_ws_all, size_t ws_bytes) {
  using namespace agq;
  if (nranks != NR || (unsigned)rank >= (unsigned)NR || n_in != 2 || ws_bytes < WS_NEED ||
      in_sizes[0] != MSH * KD || in_sizes[1] != KD * NSH || out_size != NR * MSH * NSH) {
    fprintf(stderr, "ag_gemm_q8: unexpected problem shape\n");
    return;
  }
  static bool configured = false;
  if (!configured) {
    int dev = 0, cus = 0;
    HIP_CHECK(hipGetDevice(&dev));
    HIP_CHECK(hipDeviceGetAttribute(&cus, hipDeviceAttributeMultiprocessorCount, dev));
    if (cus < NWG) fprintf(stderr, "ag_gemm_q8: needs %d CUs, found %d\n", NWG, cus);
    HIP_CHECK(hipFuncSetAttribute((const void*)ag_gemm_q8, hipFuncAttributeMaxDynamicSharedMemorySize, LDS_BYTES));
    configured = true;
  }
  Params p;
  p.x = (const char*)d_in[0];
  p.w = (const char*)d_in[1];
  p.out = (char*)d_out;
  for (int i = 0; i < NR; ++i) p.ws[i] = (char*)d_ws_all[i];
  p.myws = (char*)d_ws_all[rank];
  p.rank = rank;
  hipLaunchKernelGGL(ag_gemm_q8, dim3(NWG), dim3(NTHR), LDS_BYTES, stream, p);
  HIP_CHECK(hipGetLastError());
}
```
